```python
import numpy as np

kernel_name = "gpu_gemm_ar_m4096_k4096_n4096_e4m3e5m2_dequant"


ABSMAX_RTOL = 0.05

COMPUTE_DTYPE = "float32"
EPILOGUE = "dequant"
IN_QUANT = None
OUT_QUANT = None
IN_FP8 = "e4m3e5m2"
IN_DTYPES = {"x": "e4m3", "w_mat": "e5m2",
             "scale_x": "float32", "scale_w": "float32"}
WORLD_SIZE = 8

def _snap_fp8(x, mant_bits, max_abs, min_normal):
    a = np.abs(x.astype(np.float64))
    _m, _e = np.frexp(a)
    q = np.where(a >= min_normal, np.ldexp(1.0, _e - 1 - mant_bits),
                 min_normal * 2.0 ** -mant_bits)
    snapped = np.minimum(np.round(a / q) * q, max_abs)
    return (np.sign(x) * snapped).astype(np.float32)

def setup_inputs():

    x = _snap_fp8(np.random.randn(4096, 4096).astype(np.float32), 3, 448.0, 2.0 ** -6)
    w_mat = _snap_fp8(np.random.randn(4096, 4096).astype(np.float32), 2, 57344.0, 2.0 ** -14)
    scale_x = (0.02 * (1.0 + np.random.rand(1))).astype(np.float32)
    scale_w = (0.02 * (1.0 + np.random.rand(1))).astype(np.float32)
    return {"x": x, "w_mat": w_mat, "scale_x": scale_x, "scale_w": scale_w}

def reference(x, w_mat, scale_x, scale_w):
    acc = (x @ w_mat).astype(np.float32)
    y = (acc * (scale_x * scale_w)).astype(np.float32)
    return y
```

```cpp
#include <hip/hip_runtime.h>
#include <rccl/rccl.h>
#include <cstdio>

#define HIP_CHECK(expr) do { hipError_t _e = (expr); if (_e != hipSuccess) { \
    fprintf(stderr, "HIP error %s at %s:%d\n", hipGetErrorString(_e), __FILE__, __LINE__); \
    return; } } while (0)

#define LAS __attribute__((address_space(3)))
typedef unsigned u32;
typedef unsigned long long u64;
typedef unsigned u32x4 __attribute__((ext_vector_type(4)));
typedef unsigned u32x2 __attribute__((ext_vector_type(2)));
typedef int i32x8 __attribute__((ext_vector_type(8)));
typedef int i32x4 __attribute__((ext_vector_type(4)));
typedef float f32x16 __attribute__((ext_vector_type(16)));
typedef float f32x4 __attribute__((ext_vector_type(4)));

constexpr int NG = 8;
constexpr int MM = 4096, NN = 4096, KL = 512;
constexpr int UNIT = 16384;
constexpr int PIECE = 2048;
constexpr int NSTEP = 64;
constexpr int NSLOT = 4, SLOT = 2 * UNIT;
constexpr int L_CTRL = NSLOT * SLOT;
constexpr int L_STG = L_CTRL + 256;
constexpr int LDS_BYTES = L_STG + 8 * PIECE;
constexpr size_t W_DATA = 0;
constexpr size_t DATA_PAR = (size_t)NSTEP * 32 * UNIT;
constexpr size_t W_READY = 2 * DATA_PAR;
constexpr size_t W_CNT = W_READY + 32 * 64 * 8 * 4;
constexpr size_t W_TOK = W_CNT + 256 * 128;
constexpr size_t W_TMO = W_TOK + 256 * 128;
constexpr size_t W_END = W_TMO + 128;
constexpr u64 DEADLINE = 50000000ull;
#define PACE_DT 920
#define TOK_WN 96

struct Params {
  char* ws[NG];
  const unsigned char* x;
  const unsigned char* w;
  const float* sx;
  const float* sw;
  float* out;
  u64* dbg;
  int rank;
  int pad;
};

template <int N> __device__ __forceinline__ void wait_vm() { asm volatile("s_waitcnt vmcnt(%0)" :: "i"(N) : "memory"); }
__device__ __forceinline__ u64 rt() { return __builtin_amdgcn_s_memrealtime(); }
__device__ __forceinline__ void glds4(const char* base, u32 voff, u32 ldsaddr) {
  asm volatile("s_mov_b32 m0, %2\n\ts_nop 0\n\t"
               "global_load_lds_dwordx4 %0, %1 sc0 sc1\n\t"
               "global_load_lds_dwordx4 %0, %1 offset:1024 sc0 sc1\n\t"
               "global_load_lds_dwordx4 %0, %1 offset:2048 sc0 sc1\n\t"
               "global_load_lds_dwordx4 %0, %1 offset:3072 sc0 sc1"
               :: "v"(voff), "s"(base), "s"(ldsaddr) : "memory");
}
__device__ __forceinline__ void st2_sys(char* base, u32 voff, u32x4 a, u32x4 b) {
  asm volatile("global_store_dwordx4 %0, %1, %3 sc0 sc1\n\t"
               "global_store_dwordx4 %0, %2, %3 offset:1024 sc0 sc1\n\ts_nop 1"
               :: "v"(voff), "v"(a), "v"(b), "s"(base) : "memory");
}
__device__ __forceinline__ i32x8 ldfrag(const LAS char* p, int hi_off) {
  i32x4 lo = *(const LAS i32x4*)p, hi = *(const LAS i32x4*)(p + hi_off);
  return __builtin_shufflevector(lo, hi, 0, 1, 2, 3, 4, 5, 6, 7);
}
#define MFMA(a, b, c) __builtin_amdgcn_mfma_scale_f32_32x32x64_f8f6f4(a, b, c, 0, 1, 0, 0, 0, 0)
#define SB __builtin_amdgcn_sched_barrier(0)
#define STAMP(i) do {} while (0)

__device__ __forceinline__ void tr4(u32 r0, u32 r1, u32 r2, u32 r3, u32& c0, u32& c1, u32& c2, u32& c3) {
  u32 a = __builtin_amdgcn_perm(r1, r0, 0x05010400u);
  u32 b = __builtin_amdgcn_perm(r1, r0, 0x07030602u);
  u32 c = __builtin_amdgcn_perm(r3, r2, 0x05010400u);
  u32 d = __builtin_amdgcn_perm(r3, r2, 0x07030602u);
  c0 = __builtin_amdgcn_perm(c, a, 0x05040100u);
  c1 = __builtin_amdgcn_perm(c, a, 0x07060302u);
  c2 = __builtin_amdgcn_perm(d, b, 0x05040100u);
  c3 = __builtin_amdgcn_perm(d, b, 0x07060302u);
}

__global__ void __launch_bounds__(768, 3) fused_gemm_ag(Params p) {
  extern __shared__ __attribute__((aligned(16))) char lds_raw[];
  LAS char* lds = (LAS char*)lds_raw;
  volatile LAS u32* ctrl = (volatile LAS u32*)(lds + L_CTRL);
  const int tid = threadIdx.x, lane = tid & 63, wv = __builtin_amdgcn_readfirstlane(tid >> 6);
  const int bid = blockIdx.x, xcd = bid & 7, loc = bid >> 3;
  const int ti = (xcd >> 1) + 4 * (loc & 3), tj = (xcd & 1) + 2 * (loc >> 2);
  const int sidx = tj < ti ? ti * ti + tj : tj * tj + tj + ti;
  const int su = sidx >> 3, st = sidx & 7;
  const int rank = p.rank;
  char* const myws = p.ws[rank];
  STAMP(0);

  u32x4 q0, q1;
  if (wv < 8) {
    const int blk = su >> 1;
    if ((su & 1) == 0) {
      const unsigned char* src = p.x + (size_t)(blk * 256 + st * 32 + (lane & 31)) * KL + wv * 64 + (lane >> 5) * 32;
      q0 = *(const u32x4*)src; q1 = *(const u32x4*)(src + 16);
    } else {
      const unsigned char* src = p.w + (size_t)(wv * 64 + (st & 3) * 16 + (lane >> 5) * 8) * NN + blk * 256 + (st >> 2) * 128 + (lane & 31) * 4;
      u32 r[8];
#pragma unroll
      for (int k = 0; k < 8; ++k) r[k] = *(const u32*)(src + (size_t)k * NN);
      u32 a[4], b[4];
      tr4(r[0], r[1], r[2], r[3], a[0], a[1], a[2], a[3]);
      tr4(r[4], r[5], r[6], r[7], b[0], b[1], b[2], b[3]);
      auto s0 = __builtin_amdgcn_permlane32_swap(a[0], a[1], false, false);
      auto s1 = __builtin_amdgcn_permlane32_swap(b[0], b[1], false, false);
      auto s2 = __builtin_amdgcn_permlane32_swap(a[2], a[3], false, false);
      auto s3 = __builtin_amdgcn_permlane32_swap(b[2], b[3], false, false);
      q0 = u32x4{s0[0], s1[0], s0[1], s1[1]};
      q1 = u32x4{s2[0], s3[0], s2[1], s3[1]};
    }
    *(LAS u32x4*)(lds + L_STG + wv * PIECE + lane * 16) = q0;
    *(LAS u32x4*)(lds + L_STG + wv * PIECE + 1024 + lane * 16) = q1;
  } else if (wv == 11) {
    u32* cnt = (u32*)(myws + W_CNT + bid * 128);
    u32 c = __hip_atomic_load(cnt, __ATOMIC_RELAXED, __HIP_MEMORY_SCOPE_AGENT) + 1;
    if (lane == 0) __hip_atomic_store(cnt, c, __ATOMIC_RELAXED, __HIP_MEMORY_SCOPE_AGENT);
    if (lane < 16) ctrl[lane] = lane == 3 ? c : 0;
  }
  __syncthreads();
  const u32 call = __builtin_amdgcn_readfirstlane(ctrl[3]);
  const size_t par = (call & 1) * DATA_PAR;
  STAMP(1);

  if (wv < 8) {
    st2_sys(myws + W_DATA + par + (size_t)(wv * 32 + su) * UNIT + st * PIECE, lane * 16, q0, q1);
    wait_vm<0>();
    if (lane == 0)
      __hip_atomic_store((u32*)(myws + W_READY) + (su * 64 + wv) * 8 + st, call, __ATOMIC_RELAXED, __HIP_MEMORY_SCOPE_SYSTEM);

    const int wm = wv & 3, wn = wv >> 2;
    f32x16 acc[2][4];
#pragma unroll
    for (int a = 0; a < 2; ++a)
#pragma unroll
      for (int j = 0; j < 4; ++j)
#pragma unroll
        for (int r = 0; r < 16; ++r) acc[a][j][r] = 0.f;
    const u32 aoff = wm * 4096 + lane * 16;
    const u32 boff = UNIT + wn * 8192 + (lane >> 5) * 4096 + (lane & 31) * 16;
    for (int s = 0; s < NSTEP; ++s) {
      const u32 sb = (s & (NSLOT - 1)) * SLOT;
      while (true) {
        u32x2 f = *(volatile LAS u32x2*)(lds + L_CTRL);
        u32 m = __builtin_amdgcn_readfirstlane(f.x < f.y ? f.x : f.y);
        if (m > (u32)s) break;
        __builtin_amdgcn_s_sleep(1);
      }
      __builtin_amdgcn_fence(__ATOMIC_ACQUIRE, "workgroup");
      const LAS char* ap = lds + sb + aoff;
      const LAS char* bp = lds + sb + boff;
      i32x8 a0 = ldfrag(ap, 1024), b0 = ldfrag(bp, 2048), a1 = ldfrag(ap + 2048, 1024), b1 = ldfrag(bp + 512, 2048);
      SB;
      __builtin_amdgcn_s_setprio(1);
      acc[0][0] = MFMA(a0, b0, acc[0][0]);
      acc[1][0] = MFMA(a1, b0, acc[1][0]);
      SB;
      b0 = ldfrag(bp + 1024, 2048);
      SB;
      acc[0][1] = MFMA(a0, b1, acc[0][1]);
      acc[1][1] = MFMA(a1, b1, acc[1][1]);
      SB;
      b1 = ldfrag(bp + 1536, 2048);
      SB;
      acc[0][2] = MFMA(a0, b0, acc[0][2]);
      acc[1][2] = MFMA(a1, b0, acc[1][2]);
      SB;
      asm volatile("s_waitcnt lgkmcnt(0)" ::: "memory");
      ctrl[8 + wv] = s + 1;
      asm volatile("" : "+v"(b1) :: "memory");
      SB;
      acc[0][3] = MFMA(a0, b1, acc[0][3]);
      acc[1][3] = MFMA(a1, b1, acc[1][3]);
      SB;
      __builtin_amdgcn_s_setprio(0);
    }
    STAMP(2);
    float scale = p.sx[0] * p.sw[0];
    if (ctrl[2] != 0) scale = __builtin_nanf("");
    float* o = p.out + (size_t)(ti * 256 + wm * 64 + 4 * (lane >> 5)) * NN + tj * 256 + wn * 128 + 4 * (lane & 31);
#pragma unroll
    for (int a = 0; a < 2; ++a)
#pragma unroll
      for (int r = 0; r < 16; ++r) {
        f32x4 v = {acc[a][0][r] * scale, acc[a][1][r] * scale, acc[a][2][r] * scale, acc[a][3][r] * scale};
        *(f32x4*)(o + (size_t)(a * 32 + (r & 3) + 8 * (r >> 2)) * NN) = v;
      }
    STAMP(3);
  } else if (wv < 10) {
    const int which = wv - 8;
    const int u = which == 0 ? 2 * ti : 2 * tj + 1;
    const char* src = myws + W_DATA + par + (size_t)u * UNIT;
    const u32* rdy = (const u32*)(myws + W_READY) + u * 64 * 8;
    u32* tmo = (u32*)(myws + W_TMO);
    const u32 ldsb = (u32)(size_t)lds + which * UNIT;
    int pub = 0, known = 0;
    bool dead = false;
    for (int s = 0; s < NSTEP; ++s) {
      if (known <= s && !dead) {
        const u64 t0 = rt();
        while (true) {
          int idx = s * 8 + lane; idx = idx < 512 ? idx : 511;
          u32 v = __hip_atomic_load(rdy + idx, __ATOMIC_RELAXED, __HIP_MEMORY_SCOPE_SYSTEM);
          u64 bad = __ballot(v < call);
          if (pub < s) { wait_vm<0>(); pub = s; ctrl[which] = pub; }
          int n = bad ? (__builtin_ctzll(bad) >> 3) : 8;
          if (n > 0) { known = s + n; break; }
          if (__hip_atomic_load(tmo, __ATOMIC_RELAXED, __HIP_MEMORY_SCOPE_AGENT) == call || rt() - t0 > DEADLINE) {
            if (lane == 0) __hip_atomic_store(tmo, call, __ATOMIC_RELAXED, __HIP_MEMORY_SCOPE_AGENT);
            dead = true; ctrl[2] = 1;
            break;
          }
          __builtin_amdgcn_s_sleep(2);
        }
      }
      if (s >= NSLOT) {
        const u32 need = s - NSLOT + 1;
        u32 v = ctrl[8 + (lane & 7)];
        if (!__all(v >= need)) {
          if (pub < s) { wait_vm<0>(); pub = s; ctrl[which] = pub; }
          do { __builtin_amdgcn_s_sleep(1); v = ctrl[8 + (lane & 7)]; } while (!__all(v >= need));
        }
        __builtin_amdgcn_fence(__ATOMIC_ACQUIRE, "workgroup");
      }
      if (!dead) {
        const char* g = src + (size_t)s * 32 * UNIT;
        const u32 l = ldsb + (s & (NSLOT - 1)) * SLOT;
#pragma unroll
        for (int j = 0; j < 4; ++j) glds4(g + j * 4096, lane * 16, l + j * 4096);
      }
      if (s + 1 - pub > 2) { wait_vm<32>(); pub = s - 1; ctrl[which] = pub; }
    }
    STAMP(2);
    if (pub < NSTEP - 1) { wait_vm<16>(); ctrl[which] = NSTEP - 1; }
    if (__hip_atomic_load(tmo, __ATOMIC_RELAXED, __HIP_MEMORY_SCOPE_AGENT) == call) ctrl[2] = 1;
    wait_vm<0>();
    asm volatile("s_waitcnt lgkmcnt(0)" ::: "memory");
    ctrl[which] = NSTEP;
    STAMP(3);
  } else if (wv == 10) {
    const u64 t0 = rt();
    u32* tok = (u32*)(myws + W_TOK);
    const u32 tb = call * 2048u;
    const int fq = lane < 7 ? lane : 0;
    const int fpeer = (rank + 1 + fq) & 7;
    u32* fptr = (u32*)(p.ws[fpeer] + W_READY) + (su * 64 + 8 + ((rank - fpeer - 1) & 7)) * 8 + st;
    asm volatile("" : "+v"(fptr));
    char* dst[7];
#pragma unroll
    for (int q = 0; q < 7; ++q) {
      int qq = q + sidx % 7; qq = qq >= 7 ? qq - 7 : qq;
      const int peer = (rank + 1 + qq) & 7;
      dst[q] = p.ws[peer] + W_DATA + par + (size_t)((8 + ((rank - peer - 1) & 7)) * 32 + su) * UNIT + st * PIECE;
      asm volatile("" : "+s"(dst[q]));
    }
#pragma unroll 1
    for (int c = 0; c < 8; ++c) {
      const u32 pc = c * 256 + sidx;
      u32x4 d0 = *(const LAS u32x4*)(lds + L_STG + c * PIECE + lane * 16);
      u32x4 d1 = *(const LAS u32x4*)(lds + L_STG + c * PIECE + 1024 + lane * 16);
      const u64 due = t0 + (((u64)pc * PACE_DT) >> 8);
      while (rt() < due) __builtin_amdgcn_s_sleep(1);
      if (TOK_WN > 0 && pc >= (u32)TOK_WN) {
        const u64 tw = rt();
        while ((int)(__hip_atomic_load(tok + sidx * 32, __ATOMIC_RELAXED, __HIP_MEMORY_SCOPE_AGENT) - (tb + pc)) < 0) {
          if (rt() - tw > 2000ull) break;
          __builtin_amdgcn_s_sleep(2);
        }
      }
      const u32 voff = c * (7 * 32 * UNIT) + lane * 16;
#pragma unroll
      for (int q = 0; q < 7; ++q) st2_sys(dst[q], voff, d0, d1);
      wait_vm<0>();
      if (lane < 7) __hip_atomic_store(fptr + c * 7 * 8, call, __ATOMIC_RELAXED, __HIP_MEMORY_SCOPE_SYSTEM);
      if (TOK_WN > 0 && pc + TOK_WN < 2048u && lane == 0)
        __hip_atomic_store(tok + ((pc + TOK_WN) & 255) * 32, tb + pc + TOK_WN, __ATOMIC_RELAXED, __HIP_MEMORY_SCOPE_AGENT);
      STAMP(2 + c);
    }
  }
}

extern "C" void kernel_launch(int rank, int nranks, ncclComm_t comm,
                              hipStream_t stream,
                              void* const* d_in, const int* in_sizes, int n_in,
                              void* d_out, int out_size,
                              const int* in_dtypes, int out_dtype,
                              void* const* d_out_all,
                              void* const* d_ws_all, size_t ws_bytes) {
  if (nranks != NG || (unsigned)rank >= (unsigned)NG || n_in < 4 || in_sizes[0] != MM * KL || in_sizes[1] != KL * NN ||
      out_size != MM * NN || ws_bytes < W_END) {
    fprintf(stderr, "kernel_launch: unexpected problem shape\n");
    return;
  }
  HIP_CHECK(hipFuncSetAttribute((const void*)fused_gemm_ag, hipFuncAttributeMaxDynamicSharedMemorySize, LDS_BYTES));
  Params p;
  for (int i = 0; i < NG; ++i) p.ws[i] = (char*)d_ws_all[i];
  p.x = (const unsigned char*)d_in[0];
  p.w = (const unsigned char*)d_in[1];
  p.sx = (const float*)d_in[2];
  p.sw = (const float*)d_in[3];
  p.out = (float*)d_out;
  p.dbg = nullptr;
  p.rank = rank;
  p.pad = 0;
  hipLaunchKernelGGL(fused_gemm_ag, dim3(256), dim3(768), LDS_BYTES, stream, p);
  HIP_CHECK(hipGetLastError());
}
```

```cpp
#include <hip/hip_runtime.h>
#include <rccl/rccl.h>
#include <cstdio>

#define HIP_CHECK(expr) do { hipError_t _e = (expr); if (_e != hipSuccess) { \
    fprintf(stderr, "HIP error %s at %s:%d\n", hipGetErrorString(_e), __FILE__, __LINE__); \
    return; } } while (0)

#define LAS __attribute__((address_space(3)))
#define GAS __attribute__((address_space(1)))
typedef unsigned u32;
typedef unsigned long long u64;
typedef unsigned u32x4 __attribute__((ext_vector_type(4)));
typedef unsigned u32x2 __attribute__((ext_vector_type(2)));
typedef int i32x8 __attribute__((ext_vector_type(8)));
typedef int i32x4 __attribute__((ext_vector_type(4)));
typedef float f32x16 __attribute__((ext_vector_type(16)));
typedef float f32x4 __attribute__((ext_vector_type(4)));

constexpr int NG = 8;
constexpr int MM = 4096, NN = 4096, KL = 512;
constexpr int UNIT = 16384;
constexpr int PIECE = 2048;
constexpr int NSTEP = 64;
constexpr int NSLOT = 4, SLOT = 2 * UNIT;
constexpr int L_CTRL = NSLOT * SLOT;
constexpr int L_LUTC = L_CTRL + 256;
constexpr int L_LUTD = L_LUTC + 256;
constexpr int L_RDY = L_LUTD + 1024;
constexpr int L_DEC = L_RDY + 512;
constexpr int DEC_BYTES = 512 + 1024 + 2 * PIECE;
constexpr int L_STG = L_DEC + 2 * DEC_BYTES;
constexpr int LDS_BYTES = L_STG + 8 * PIECE;
static_assert(LDS_BYTES <= 160 * 1024, "LDS");
#define NT 32
constexpr u32 NB = 256 - NT;
constexpr u32 NDC = 36;
constexpr int CSTRIDE = UNIT + 256;
constexpr size_t W_DATA = 0;
constexpr size_t DATA_PAR = (size_t)NSTEP * 32 * UNIT;
constexpr size_t W_CDATA = 2 * DATA_PAR;
constexpr size_t CDATA_PAR = (size_t)256 * 7 * CSTRIDE;
constexpr size_t W_CFLAG = W_CDATA + 2 * CDATA_PAR;
constexpr size_t CFLAG_PAR = 256 * 8 * 128;
constexpr size_t W_READY = W_CFLAG + 2 * CFLAG_PAR;
constexpr size_t W_UCNT = W_READY + 32 * 64 * 128;
constexpr size_t W_SCNT = W_UCNT + 256 * 64;
constexpr size_t W_CNT = W_SCNT + 256 * 64;
constexpr size_t W_TMO = W_CNT + 256 * 128;
constexpr size_t W_END = W_TMO + 128;
constexpr u64 DEADLINE = 50000000ull;
#define DT_RAW 940
#define DT_PK 735
#define ILV 13
#define PAUSE_LEAD 20
#define TGAP 0
#define LSLEEP 1
#define DSLEEP 4
#define PWIN 14
#define POLLP 30
#define PACE_T0 90
constexpr u32 ST_RAW = 1u << 12, ST_VALID = 1u << 31;

struct Params {
  char* ws[NG];
  const unsigned char* x;
  const unsigned char* w;
  const float* sx;
  const float* sw;
  float* out;
  u64* dbg;
  int rank;
  int pad;
};

#define GPOL " sc0 sc1"
template <int N> __device__ __forceinline__ void wait_vm() { asm volatile("s_waitcnt vmcnt(%0)" :: "i"(N) : "memory"); }
__device__ __forceinline__ u32 vmcnt_now() {
  u32 r;
  asm volatile("s_getreg_b32 %0, hwreg(HW_REG_IB_STS)" : "=s"(r) :: "memory");
  return (r & 15) | ((r >> 18) & 0x30);
}
__device__ __forceinline__ u64 rt() { return __builtin_amdgcn_s_memrealtime(); }
__device__ __forceinline__ void glds4(const char* base, u32 voff, u32 ldsaddr) {
  asm volatile("s_mov_b32 m0, %2\n\ts_nop 0\n\t"
               "global_load_lds_dwordx4 %0, %1" GPOL "\n\t"
               "global_load_lds_dwordx4 %0, %1 offset:1024" GPOL "\n\t"
               "global_load_lds_dwordx4 %0, %1 offset:2048" GPOL "\n\t"
               "global_load_lds_dwordx4 %0, %1 offset:3072" GPOL
               :: "v"(voff), "s"(base), "s"(ldsaddr) : "memory");
}
__device__ __forceinline__ void st2_sys(char* base, u32 voff, u32x4 a, u32x4 b) {
  asm volatile("global_store_dwordx4 %0, %1, %3 sc0 sc1\n\t"
               "global_store_dwordx4 %0, %2, %3 offset:1024 sc0 sc1\n\ts_nop 1"
               :: "v"(voff), "v"(a), "v"(b), "s"(base) : "memory");
}
__device__ __forceinline__ i32x8 ldfrag(const LAS char* p, int hi_off) {
  i32x4 lo = *(const LAS i32x4*)p, hi = *(const LAS i32x4*)(p + hi_off);
  return __builtin_shufflevector(lo, hi, 0, 1, 2, 3, 4, 5, 6, 7);
}
#define MFMA(a, b, c) __builtin_amdgcn_mfma_scale_f32_32x32x64_f8f6f4(a, b, c, 0, 1, 0, 0, 0, 0)
#define SB __builtin_amdgcn_sched_barrier(0)
#define STAMP(i) do {} while (0)

__device__ __forceinline__ void tr4(u32 r0, u32 r1, u32 r2, u32 r3, u32& c0, u32& c1, u32& c2, u32& c3) {
  u32 a = __builtin_amdgcn_perm(r1, r0, 0x05010400u);
  u32 b = __builtin_amdgcn_perm(r1, r0, 0x07030602u);
  u32 c = __builtin_amdgcn_perm(r3, r2, 0x05010400u);
  u32 d = __builtin_amdgcn_perm(r3, r2, 0x07030602u);
  c0 = __builtin_amdgcn_perm(c, a, 0x05040100u);
  c1 = __builtin_amdgcn_perm(c, a, 0x07060302u);
  c2 = __builtin_amdgcn_perm(d, b, 0x05040100u);
  c3 = __builtin_amdgcn_perm(d, b, 0x07060302u);
}

#define CFENCE() asm volatile("" ::: "memory")
__device__ __forceinline__ u32 rotr32(u32 x, u32 n) { return __builtin_amdgcn_alignbit(x, x, n); }
__device__ __forceinline__ u32 bfi(u32 m, u32 a, u32 b) { return (a & m) | (b & ~m); }
__device__ __forceinline__ u32 wave_incl_scan(u32 v) {
  v += (u32)__builtin_amdgcn_update_dpp(0, (int)v, 0x111, 0xf, 0xf, true);
  v += (u32)__builtin_amdgcn_update_dpp(0, (int)v, 0x112, 0xf, 0xf, true);
  v += (u32)__builtin_amdgcn_update_dpp(0, (int)v, 0x114, 0xf, 0xf, true);
  v += (u32)__builtin_amdgcn_update_dpp(0, (int)v, 0x118, 0xf, 0xf, true);
  v += (u32)__builtin_amdgcn_update_dpp(0, (int)v, 0x142, 0xa, 0xf, false);
  v += (u32)__builtin_amdgcn_update_dpp(0, (int)v, 0x143, 0xc, 0xf, false);
  return v;
}
__device__ __forceinline__ u32 lutd_entry(u32 idx) {
  u32 s = 0, r = 0;
#pragma unroll
  for (int i = 0; i < 4; ++i) { u32 c = (idx >> (2 * i)) & 3; s |= (c == 3 ? r : 4 + c) << (8 * i); r += c == 3; }
  return s;
}
__device__ __forceinline__ u32x4 lutc_entry(u32 pat) {
  u32 s = 0x0c0c0c0cu, r = 0;
#pragma unroll
  for (int i = 0; i < 4; ++i) if ((pat >> i) & 1) { s = (s & ~(0xffu << (8 * r))) | ((u32)i << (8 * r)); ++r; }
  u32 sw = __builtin_amdgcn_perm(s, s, 0x02030001u);
  return u32x4{s, 2 * r, sw, 4 * r};
}
__device__ __forceinline__ void lds_or_bits(LAS char* base, u32 bitpos, u32 x) {
  u64 y = (u64)x << (bitpos & 31);
  LAS u32* q = (LAS u32*)(base + ((bitpos >> 5) << 2));
  __hip_atomic_fetch_or(q, (u32)y, __ATOMIC_RELAXED, __HIP_MEMORY_SCOPE_WORKGROUP);
  __hip_atomic_fetch_or(q + 1, (u32)(y >> 32), __ATOMIC_RELAXED, __HIP_MEMORY_SCOPE_WORKGROUP);
}
template <bool ISW>
__device__ __forceinline__ u32 encode_piece(const u32x4& q0, const u32x4& q1, LAS char* msg, const LAS char* lutc, int lane, u32& hdr) {
  constexpr u32 MB = ISW ? 2 : 3, EM1 = ISW ? 31 : 15, EM = EM1 * 0x01010101u, TAIL = ISW ? 1280 : 1536;
  const u32 v[8] = {q0.x, q0.y, q0.z, q0.w, q1.x, q1.y, q1.z, q1.w};
  u32 ex[8], sum = 0;
#pragma unroll
  for (int j = 0; j < 8; ++j) { ex[j] = (v[j] >> MB) & EM; sum = __builtin_amdgcn_sad_u8(ex[j], 0, sum); }
  *(LAS u32x4*)(msg + 1024 + lane * 16) = u32x4{0, 0, 0, 0};
  const u32 tot = __builtin_amdgcn_readlane(wave_incl_scan(sum), 63);
  const u32 c = (tot + 1638) >> 11;
  const u32 kadd = ((EM1 + 1 + 3 - c + EM1 + 1) & EM1) * 0x01010101u;
  constexpr u32 TL = 0x10003534u, TH = 0x3f3f3620u;
  u32 l1[2], a2[2], a3[2], p2[2], p3[2], bad = 0;
#pragma unroll
  for (int w = 0; w < 2; ++w) {
    l1[w] = 0; a2[w] = 0; a3[w] = 0; p2[w] = 0; p3[w] = 0;
#pragma unroll
    for (int k = 0; k < 4; ++k) {
      const u32 r = (ex[4 * w + k] + kadd) & EM;
      const u32 t8 = r + 0x78787878u;
      const u32 sel = (t8 & 0x80808080u) | r;
      const u32 cc = __builtin_amdgcn_perm(TH, TL, sel);
      const u32x2 e1 = *(const LAS u32x2*)(lutc + __builtin_amdgcn_udot4(cc & 0x04040404u, 0x20100804u, 0, false));
      const u32x2 e2 = *(const LAS u32x2*)(lutc + 8 + __builtin_amdgcn_udot4(cc & 0x08080808u, 0x10080402u, 0, false));
      const u32 i1 = __builtin_amdgcn_udot4(cc & 0x30303030u, 0x40100401u, 0, false);
      l1[w] |= k == 0 ? i1 >> 4 : i1 << (8 * k - 4);
      const u32 c2 = __builtin_amdgcn_udot4(__builtin_amdgcn_perm(cc, cc, e1.x) & 0x03030303u, 0x40100401u, 0, false);
      a2[w] |= c2 << p2[w]; p2[w] += e1.y;
      u32 nib = r;
      if (ISW) { nib = r & __builtin_amdgcn_perm(0x01010101u, 0x01010101u, sel) & 0x0f0f0f0fu; bad |= t8 & ~(r + 0x6e6e6e6eu); }
      const u32 x = __builtin_amdgcn_perm(nib, nib, e2.x);
      const u32 y = __builtin_amdgcn_perm(0, (x << 12) | x, 0x0c0c0301u);
      a3[w] |= y << (p3[w] & 31); p3[w] += e2.y;
    }
  }
  const bool lbad = (bad & 0x80808080u) != 0 || p3[0] > 32 || p3[1] > 32;
  const u32 n = (p2[0] + p2[1]) | ((p3[0] + p3[1]) << 16);
  const u32 inc = wave_incl_scan(n);
  const u32 totn = __builtin_amdgcn_readlane(inc, 63);
  const u32 w2 = ((totn & 0xffff) + 31) >> 5, w3 = ((totn >> 16) + 31) >> 5;
  const u32 bytes = (TAIL + 4 * (w2 + w3) + 31) & ~31u;
  if (__any(lbad) || w2 > 64 || w3 > 64 || bytes > 2016) return 0;
  u32 sm[4];
  if (!ISW) {
#pragma unroll
    for (int k = 0; k < 4; ++k) sm[k] = bfi(0x87878787u, v[2 * k], rotr32(v[2 * k + 1], 28));
    *(LAS u32x2*)(msg + 1024 + lane * 8) = u32x2{sm[2], sm[3]};
  } else {
#pragma unroll
    for (int k = 0; k < 3; ++k) sm[k] = bfi(0x83838383u, v[2 * k], rotr32(v[2 * k + 1], 29));
    sm[0] = bfi(0x60606060u, v[6] << 5, sm[0]);
    sm[1] = bfi(0x60606060u, v[7] << 5, sm[1]);
    sm[2] = bfi(0x60606060u, bfi(0x20202020u, v[6] >> 2, v[7] >> 1), sm[2]);
    *(LAS u32*)(msg + 1024 + lane * 4) = sm[2];
  }
  *(LAS u32x4*)(msg + lane * 16) = u32x4{l1[0], l1[1], sm[0], sm[1]};
  const u32 exc = inc - n;
  const u32 b2 = exc & 0xffff, b3 = (exc >> 16) + 32 * w2;
  lds_or_bits(msg + TAIL, b2, a2[0]);
  lds_or_bits(msg + TAIL, b2 + p2[0], a2[1]);
  lds_or_bits(msg + TAIL, b3, a3[0]);
  lds_or_bits(msg + TAIL, b3 + p3[0], a3[1]);
  hdr = (c & 31) | (w2 << 5);
  return bytes;
}
__device__ __forceinline__ u32 esc_bits(u32 w) { return w & (w >> 1) & 0x55555555u; }
__device__ __forceinline__ u32 lds_ld(u32 a) { return *(const LAS u32*)(size_t)a; }
template <bool ISW>
__device__ __forceinline__ void decode_piece(const LAS char* msg, LAS char* b2, LAS char* b3, const LAS char* lutd, u32 hdr, int lane, u32x4& q0, u32x4& q1) {
  constexpr u32 MB = ISW ? 2 : 3, EM1 = ISW ? 31 : 15, EM = EM1 * 0x01010101u, TAIL = ISW ? 1280 : 1536;
  const u32 base = ((hdr & 31) + EM1 + 1 - 3) & EM1, w2n = (hdr >> 5) & 0x7f;
  const u32 b4 = base * 0x01010101u;
  const u32 t1 = ((b4 + 0x00040302u) & EM) << MB, t2 = ((b4 + 0x00050100u) & EM) << MB;
  const u32x4 A = *(const LAS u32x4*)(msg + lane * 16);
  const u32 w2 = *(const LAS u32*)(msg + TAIL + lane * 4);
  const u32 w3 = *(const LAS u32*)(msg + TAIL + w2n * 4 + lane * 4);
  u32 sm2, sm3 = 0;
  if (!ISW) { const u32x2 B = *(const LAS u32x2*)(msg + 1024 + lane * 8); sm2 = B.x; sm3 = B.y; }
  else sm2 = *(const LAS u32*)(msg + 1024 + lane * 4);
  const u32 cw[3] = {w2, A.x, A.y};
  u32 sel[12];
#pragma unroll
  for (int i = 0; i < 12; ++i) sel[i] = *(const LAS u32*)(lutd + (((cw[i >> 2] >> (8 * (i & 3))) & 0xff) << 2));
  SB;
  u32 lo = w3 & 0x0f0f0f0fu, hi = (w3 >> 4) & 0x0f0f0f0fu;
  if (ISW) {
    const u32 gl = ((lo + 0x7e7e7e7eu) & 0x80808080u) >> 6, gh = ((hi + 0x7e7e7e7eu) & 0x80808080u) >> 6;
    lo += gl * 5 + 0x06060606u; hi += gh * 5 + 0x06060606u;
  }
  lo = ((lo + b4) & EM) << MB; hi = ((hi + b4) & EM) << MB;
  *(LAS u32x2*)(b3 + lane * 8) = u32x2{__builtin_amdgcn_perm(hi, lo, 0x05010400u), __builtin_amdgcn_perm(hi, lo, 0x07030602u)};
  CFENCE();
  const u32 e[3] = {esc_bits(w2), esc_bits(A.x), esc_bits(A.y)};
  const u32 n0 = __builtin_popcount(e[1]);
  const u32 n = (n0 + __builtin_popcount(e[2])) | (__builtin_popcount(e[0]) << 16);
  const u32 exc = wave_incl_scan(n) - n;
  const u32 o2 = (u32)(size_t)b2 + (exc & 0x7ff);
  const u32 ob[3] = {(u32)(size_t)b3 + ((exc >> 16) & 0x3ff), o2, o2 + n0};
  u32 ad[12], r0[12], r1[12], x[12];
#pragma unroll
  for (int i = 0; i < 12; ++i) ad[i] = ob[i >> 2] + ((i & 3) == 0 ? 0 : __builtin_popcount(e[i >> 2] << (32 - 8 * (i & 3))));
#pragma unroll
  for (int i = 0; i < 4; ++i) { r0[i] = lds_ld(ad[i] & ~3u); r1[i] = lds_ld((ad[i] & ~3u) + 4); }
  SB;
#pragma unroll
  for (int i = 0; i < 4; ++i) x[i] = __builtin_amdgcn_perm(t2, __builtin_amdgcn_alignbyte(r1[i], r0[i], ad[i]), sel[i]);
  *(LAS u32x4*)(b2 + lane * 16) = u32x4{x[0], x[1], x[2], x[3]};
  CFENCE();
#pragma unroll
  for (int i = 4; i < 12; ++i) { r0[i] = lds_ld(ad[i] & ~3u); r1[i] = lds_ld((ad[i] & ~3u) + 4); }
  SB;
#pragma unroll
  for (int i = 4; i < 12; ++i) x[i] = __builtin_amdgcn_perm(t1, __builtin_amdgcn_alignbyte(r1[i], r0[i], ad[i]), sel[i]);
  if (!ISW) {
    constexpr u32 S = 0x87878787u;
    q0 = u32x4{(A.z & S) | x[4], (rotr32(A.z, 4) & S) | x[5], (A.w & S) | x[6], (rotr32(A.w, 4) & S) | x[7]};
    q1 = u32x4{(sm2 & S) | x[8], (rotr32(sm2, 4) & S) | x[9], (sm3 & S) | x[10], (rotr32(sm3, 4) & S) | x[11]};
  } else {
    constexpr u32 S = 0x83838383u;
    q0 = u32x4{(A.z & S) | x[4], (rotr32(A.z, 3) & S) | x[5], (A.w & S) | x[6], (rotr32(A.w, 3) & S) | x[7]};
    q1 = u32x4{(sm2 & S) | x[8], (rotr32(sm2, 3) & S) | x[9],
               (bfi(0x03030303u, A.z >> 5, sm2 << 2) & S) | x[10], (bfi(0x03030303u, A.w >> 5, sm2 << 1) & S) | x[11]};
  }
  CFENCE();
}

__device__ __forceinline__ void load_piece(const Params& p, int g, int t, int lane, u32x4& q0, u32x4& q1) {
  const int c = g >> 5, blk = (g & 31) >> 1;
  if ((g & 1) == 0) {
    const unsigned char* src = p.x + (size_t)(blk * 256 + t * 32 + (lane & 31)) * KL + c * 64 + (lane >> 5) * 32;
    q0 = *(const u32x4*)src; q1 = *(const u32x4*)(src + 16);
  } else {
    const unsigned char* src = p.w + (size_t)(c * 64 + (t & 3) * 16 + (lane >> 5) * 8) * NN + blk * 256 + (t >> 2) * 128 + (lane & 31) * 4;
    u32 r[8];
#pragma unroll
    for (int k = 0; k < 8; ++k) r[k] = *(const u32*)(src + (size_t)k * NN);
    u32 a[4], b[4];
    tr4(r[0], r[1], r[2], r[3], a[0], a[1], a[2], a[3]);
    tr4(r[4], r[5], r[6], r[7], b[0], b[1], b[2], b[3]);
    auto s0 = __builtin_amdgcn_permlane32_swap(a[0], a[1], false, false);
    auto s1 = __builtin_amdgcn_permlane32_swap(b[0], b[1], false, false);
    auto s2 = __builtin_amdgcn_permlane32_swap(a[2], a[3], false, false);
    auto s3 = __builtin_amdgcn_permlane32_swap(b[2], b[3], false, false);
    q0 = u32x4{s0[0], s1[0], s0[1], s1[1]};
    q1 = u32x4{s2[0], s3[0], s2[1], s3[1]};
  }
}
__device__ __forceinline__ u32 lds_ticket(volatile LAS u32* w) {
  return __hip_atomic_fetch_add((LAS u32*)w, 1u, __ATOMIC_RELAXED, __HIP_MEMORY_SCOPE_WORKGROUP);
}

__global__ void __launch_bounds__(768, 3) fused_gemm_ag(Params p) {
  extern __shared__ __attribute__((aligned(16))) char lds_raw[];
  LAS char* lds = (LAS char*)lds_raw;
  volatile LAS u32* ctrl = (volatile LAS u32*)(lds + L_CTRL);
  const int tid = threadIdx.x, lane = tid & 63, wv = __builtin_amdgcn_readfirstlane(tid >> 6);
  const int bid = blockIdx.x, xcd = bid & 7, loc = bid >> 3;
  const int ti = (xcd >> 1) + 4 * (loc & 3), tj = (xcd & 1) + 2 * (loc >> 2);
  const int li = (xcd >> 1) + 4 * (lane & 3), lj = (xcd & 1) + 2 * ((lane & 31) >> 2);
  const int lrank = __builtin_popcount((u32)__ballot((lj < li ? li * li + lj : lj * lj + lj + li) < (tj < ti ? ti * ti + tj : tj * tj + tj + ti)));
  const int sidx = lrank * 8 + (xcd ^ (lrank & 1));
  const int gb = sidx - NT;
  const int te = sidx - (256 - 8 * NT);
  const int gt = NB + (te >> 3), tt = te & 7;
  const int rank = p.rank;
  char* const myws = p.ws[rank];
  STAMP(0);

  u32x4 q0, q1;
  if (wv < 8 ? gb >= 0 : (wv == 9 && te >= 0)) {
    load_piece(p, wv < 8 ? gb : gt, wv < 8 ? wv : tt, lane, q0, q1);
  } else if (wv == 8) {
    *(LAS u32x4*)(lds + L_LUTD + lane * 16) = u32x4{lutd_entry(4 * lane), lutd_entry(4 * lane + 1), lutd_entry(4 * lane + 2), lutd_entry(4 * lane + 3)};
  } else if (wv == 10) {
    if (lane < 16) *(LAS u32x4*)(lds + L_LUTC + lane * 16) = lutc_entry(lane);
  } else if (wv == 11) {
    u32* cnt = (u32*)(myws + W_CNT + bid * 128);
    u32 c = __hip_atomic_load(cnt, __ATOMIC_RELAXED, __HIP_MEMORY_SCOPE_AGENT) + 1;
    if (lane == 0) __hip_atomic_store(cnt, c, __ATOMIC_RELAXED, __HIP_MEMORY_SCOPE_AGENT);
    const u32 sc = __builtin_bit_cast(u32, p.sx[0] * p.sw[0]);
    ctrl[lane] = lane == 3 ? c : lane == 4 ? sc : lane == 7 ? 1 : 0;
    *(LAS u32x2*)(lds + L_RDY + lane * 8) = u32x2{0, 0};
  }
  __syncthreads();
  const u32 call = __builtin_amdgcn_readfirstlane(ctrl[3]);
  const size_t par = (call & 1) * DATA_PAR;
  u32* const tmo = (u32*)(myws + W_TMO);
  STAMP(1);
  if (wv >= 8) __builtin_amdgcn_s_setprio(3);

  if (wv < 8) {
    if (gb >= 0) {
      st2_sys(myws + W_DATA + par + (size_t)gb * UNIT + wv * PIECE, lane * 16, q0, q1);
      LAS char* msg = lds + L_STG + wv * PIECE;
      u32 bytes, hdr = 0;
      if (gb & 1) bytes = encode_piece<true>(q0, q1, msg, lds + L_LUTC, lane, hdr);
      else bytes = encode_piece<false>(q0, q1, msg, lds + L_LUTC, lane, hdr);
      if (bytes == 0) {
        *(LAS u32x4*)(msg + lane * 16) = q0;
        *(LAS u32x4*)(msg + 1024 + lane * 16) = q1;
        bytes = PIECE; hdr = ST_RAW;
      }
      asm volatile("s_waitcnt lgkmcnt(0)" ::: "memory");
      ctrl[24 + wv] = ST_VALID | (bytes << 16) | hdr;
      wait_vm<0>();
      if (lane == 0 && lds_ticket(ctrl + 5) == 7)
        __hip_atomic_store((u32*)(myws + W_READY + ((gb & 31) * 64 + (gb >> 5)) * 128), call, __ATOMIC_RELAXED, __HIP_MEMORY_SCOPE_SYSTEM);
    }

    const int wm = wv & 3, wn = wv >> 2;
    f32x16 acc[2][4];
#pragma unroll
    for (int a = 0; a < 2; ++a)
#pragma unroll
      for (int j = 0; j < 4; ++j)
#pragma unroll
        for (int r = 0; r < 16; ++r) acc[a][j][r] = 0.f;
    {
      u32 va = wm * 4096 + lane * 16;
      u32 vb = UNIT + wn * 8192 + (lane >> 5) * 4096 + (lane & 31) * 16;
      u32 vc = L_CTRL + wv * 4, vt, ss, sk;
#define FA0 "v[136:143]"
#define FA1 "v[144:151]"
#define FB0 "v[152:159]"
#define FB1 "v[160:167]"
#define MM(c, a, b) "v_mfma_f32_32x32x64_f8f6f4 " c ", " a ", " b ", " c " blgp:1\n\t"
#define RD(lo, hi, ad, o0, o1) "ds_read_b128 " lo ", " ad " offset:" #o0 "\n\tds_read_b128 " hi ", " ad " offset:" #o1 "\n\t"
#define SPIN "1:\n\t" \
        "ds_read_b32 %[vt], %[vc] offset:64\n\t" \
        "s_waitcnt lgkmcnt(0)\n\t" \
        "v_readfirstlane_b32 %[sk], %[vt]\n\t" \
        "s_cmp_gt_u32 %[sk], %[ss]\n\t" \
        "s_cbranch_scc1 2f\n\t" \
        "s_sleep 1\n\t" \
        "s_branch 1b\n\t" \
        "2:\n\t"
      asm volatile(
        "s_mov_b32 %[ss], 0\n\t"
        SPIN
        RD("v[136:139]", "v[140:143]", "%[va]", 0, 1024)
        RD("v[152:155]", "v[156:159]", "%[vb]", 0, 2048)
        RD("v[144:147]", "v[148:151]", "%[va]", 2048, 3072)
        RD("v[160:163]", "v[164:167]", "%[vb]", 512, 2560)
        "5:\n\t"
        "s_waitcnt lgkmcnt(4)\n\t"
        MM("%[c0]", FA0, FB0)
        "s_waitcnt lgkmcnt(2)\n\t"
        MM("%[c4]", FA1, FB0)
        RD("v[152:155]", "v[156:159]", "%[vb]", 1024, 3072)
        "s_waitcnt lgkmcnt(2)\n\t"
        MM("%[c1]", FA0, FB1)
        MM("%[c5]", FA1, FB1)
        RD("v[160:163]", "v[164:167]", "%[vb]", 1536, 3584)
        "ds_read_b32 %[vt], %[vc] offset:64\n\t"
        "v_add_u32 %[va], 0x8000, %[va]\n\t"
        "v_add_u32 %[vb], 0x8000, %[vb]\n\t"
        "s_add_u32 %[ss], %[ss], 1\n\t"
        "s_waitcnt lgkmcnt(3)\n\t"
        MM("%[c2]", FA0, FB0)
        "v_and_b32 %[va], 0x1ffff, %[va]\n\t"
        "v_and_b32 %[vb], 0x1ffff, %[vb]\n\t"
        "s_waitcnt lgkmcnt(0)\n\t"
        MM("%[c3]", FA0, FB1)
        "v_readfirstlane_b32 %[sk], %[vt]\n\t"
        "v_mov_b32 %[vt], %[ss]\n\t"
        "ds_write_b32 %[vc], %[vt] offset:32\n\t"
        "s_cmp_eq_u32 %[ss], 64\n\t"
        "s_cbranch_scc1 6f\n\t"
        "s_cmp_gt_u32 %[sk], %[ss]\n\t"
        "s_cbranch_scc1 2f\n\t"
        SPIN
        RD("v[136:139]", "v[140:143]", "%[va]", 0, 1024)
        MM("%[c6]", FA1, FB0)
        RD("v[152:155]", "v[156:159]", "%[vb]", 0, 2048)
        MM("%[c7]", FA1, FB1)
        RD("v[144:147]", "v[148:151]", "%[va]", 2048, 3072)
        RD("v[160:163]", "v[164:167]", "%[vb]", 512, 2560)
        "s_branch 5b\n\t"
        "6:\n\t"
        MM("%[c6]", FA1, FB0)
        MM("%[c7]", FA1, FB1)
        "s_waitcnt lgkmcnt(0)\n\t"
        "s_nop 15\n\ts_nop 7"
        : [c0] "+v"(acc[0][0]), [c1] "+v"(acc[0][1]), [c2] "+v"(acc[0][2]), [c3] "+v"(acc[0][3]),
          [c4] "+v"(acc[1][0]), [c5] "+v"(acc[1][1]), [c6] "+v"(acc[1][2]), [c7] "+v"(acc[1][3]),
          [va] "+v"(va), [vb] "+v"(vb), [vt] "=&v"(vt), [ss] "=&s"(ss), [sk] "=&s"(sk)
        : [vc] "v"(vc)
        : "memory", "scc",
          "v136","v137","v138","v139","v140","v141","v142","v143","v144","v145","v146","v147","v148","v149","v150","v151",
          "v152","v153","v154","v155","v156","v157","v158","v159","v160","v161","v162","v163","v164","v165","v166","v167");
    }
    STAMP(2);
    float scale = __builtin_bit_cast(float, ctrl[4]);
    if (ctrl[2] != 0) scale = __builtin_nanf("");
    char* ob = (char*)(p.out + (size_t)(ti * 256 + wm * 64) * NN + tj * 256 + wn * 128);
    const u32 oo = (4 * (lane >> 5) * NN + 4 * (lane & 31)) * 4;
#pragma unroll
    for (int a = 0; a < 2; ++a)
#pragma unroll
      for (int r = 0; r < 16; ++r) {
        f32x4 v = {acc[a][0][r] * scale, acc[a][1][r] * scale, acc[a][2][r] * scale, acc[a][3][r] * scale};
        char* rb = ob + (size_t)(a * 32 + (r & 3) + 8 * (r >> 2)) * NN * 4;
#define OUTPOL ""
        asm volatile("global_store_dwordx4 %0, %1, %2" OUTPOL "\n\ts_nop 1" :: "v"(oo), "v"(v), "s"(rb) : "memory");
      }
    STAMP(3);
  } else if (wv == 8) {
    const char* srcA = myws + W_DATA + par + (size_t)(2 * ti) * UNIT;
    const char* srcB = myws + W_DATA + par + (size_t)(2 * tj + 1) * UNIT;
    auto ready = [&]() -> u64 {
      const u32x2 f = *(volatile LAS u32x2*)(lds + L_RDY + lane * 8);
      return __ballot(f.x >= call && f.y >= call);
    };
    u64 todo = ~0ull, avail = 0;
    int pub = 0;
    bool dead = false;
#define LAP(x) do {} while (0)
    for (int s = 0; s < NSTEP; ++s) {
      avail &= todo;
      if (avail == 0 && !dead) {
        const u64 t0 = rt();
        for (int it = 1;; ++it) {
          avail = ready() & todo;
          if (avail) break;
          if (pub < s && vmcnt_now() == 0) { wait_vm<0>(); pub = s; ctrl[16 + (lane & 7)] = pub; }
          if ((it & 1023) == 0 && (__hip_atomic_load(tmo, __ATOMIC_RELAXED, __HIP_MEMORY_SCOPE_AGENT) == call || rt() - t0 > DEADLINE)) {
            if (lane == 0) __hip_atomic_store(tmo, call, __ATOMIC_RELAXED, __HIP_MEMORY_SCOPE_AGENT);
            dead = true; ctrl[2] = 1;
            break;
          }
          __builtin_amdgcn_s_sleep(LSLEEP);
        }
      }
      LAP(tA);
      {
        const u32 need = lane == 0 ? 1 : s >= NSLOT ? s - NSLOT + 1 : 0;
        while (!__all(ctrl[lane == 0 ? 7 : 8 + (lane & 7)] >= need)) {
          if (pub < s && vmcnt_now() == 0) { wait_vm<0>(); pub = s; ctrl[16 + (lane & 7)] = pub; }
          __builtin_amdgcn_s_sleep(1);
        }
      }
      __builtin_amdgcn_fence(__ATOMIC_ACQUIRE, "workgroup");
      LAP(tB);
      if (dead) { wait_vm<0>(); pub = s; ctrl[16 + (lane & 7)] = pub; continue; }
      const int pos = __builtin_ctzll(avail);
      todo &= ~(1ull << pos);
      const size_t go = (size_t)pos * 32 * UNIT;
      const u32 l = (s & (NSLOT - 1)) * SLOT;
#pragma unroll
      for (int j = 0; j < 4; ++j) glds4(srcA + go + j * 4096, lane * 16, l + j * 4096);
#pragma unroll
      for (int j = 0; j < 3; ++j) glds4(srcB + go + j * 4096, lane * 16, l + UNIT + j * 4096);
      if (pub < s) { wait_vm<28>(); pub = s; ctrl[16 + (lane & 7)] = pub; }
      glds4(srcB + go + 3 * 4096, lane * 16, l + UNIT + 3 * 4096);
      LAP(tC);
    }
    STAMP(2);
    if (__hip_atomic_load(tmo, __ATOMIC_RELAXED, __HIP_MEMORY_SCOPE_AGENT) == call) ctrl[2] = 1;
    wait_vm<0>();
    asm volatile("s_waitcnt lgkmcnt(0)" ::: "memory");
    ctrl[16 + (lane & 7)] = NSTEP;
    STAMP(3);
  } else if (wv == 9) {
    const u64 t0 = rt();
    const int fq = lane < 14 ? lane >> 1 : 0;
    char* fws = p.ws[(rank + 1 + fq) & 7];
    asm volatile("" : "+v"(fws));
    char* dst[7];
    const int rot = sidx % 7;
#pragma unroll
    for (int q = 0; q < 7; ++q) {
      int qq = q + rot; qq = qq >= 7 ? qq - 7 : qq;
      dst[q] = p.ws[(rank + 1 + qq) & 7];
      asm volatile("" : "+s"(dst[q]));
    }
    auto send = [&](size_t o0, u32 o1, const u32x4& d0, const u32x4& d1, u64 sm) {
#pragma unroll
      for (int q = 0; q < 7; ++q) {
        int qq = q + rot; qq = qq >= 7 ? qq - 7 : qq;
        char* b = dst[q] + o0 + (u32)(6 - qq) * o1;
        asm volatile("global_store_dwordx4 %0, %1, %3 sc0 sc1\n\t"
                     "s_mov_b64 exec, %4\n\t"
                     "global_store_dwordx4 %0, %2, %3 offset:1024 sc0 sc1\n\t"
                     "s_mov_b64 exec, -1\n\ts_nop 1"
                     :: "v"(lane * 16), "v"(d0), "v"(d1), "s"(b), "s"(sm) : "memory");
      }
    };
    if (te >= 0) {
      st2_sys(myws + W_DATA + par + (size_t)gt * UNIT + tt * PIECE, lane * 16, q0, q1);
      wait_vm<0>();
      if (lane == 0) {
        const u32 old = __hip_atomic_fetch_add((u32*)(myws + W_UCNT + gt * 64), 1u, __ATOMIC_RELAXED, __HIP_MEMORY_SCOPE_AGENT);
        if ((old & 7) == 7) __hip_atomic_store((u32*)(myws + W_READY + ((gt & 31) * 64 + (gt >> 5)) * 128), call, __ATOMIC_RELAXED, __HIP_MEMORY_SCOPE_SYSTEM);
      }
    }
    const char* rbase = myws + W_READY;
    const u32 rl = (((lane & 1) ? 2 * tj + 1 : 2 * ti) * 64 + (lane >> 1)) * 128;
    int slot = 0;
#pragma unroll
    for (int t = 0; t < 8; ++t) slot += min(max((8 * gb + ILV * ((lane & 7) - t) + 7) >> 3, 0), (int)NB);
    const u32 dueo = PACE_T0 + ((slot * (u32)DT_PK) >> 8);
    const u64 dueb = t0 + __builtin_amdgcn_readfirstlane(dueo);
    const u64 duet = t0 + PACE_T0 + TGAP + ((NB * (u32)(8 * DT_PK) + te * (u32)DT_RAW) >> 8);
    bool sendb = gb >= 0, sendt = te >= 0, allseen = false;
    u64 nextp = t0;
    while (true) {
      const u64 now = rt();
      if (sendb && now + PAUSE_LEAD >= dueb) {
        ctrl[7] = 0;
        const size_t o0 = W_CDATA + (call & 1) * CDATA_PAR + (size_t)gb * 7 * CSTRIDE;
#pragma unroll 1
        for (int t = 0; t < 8; ++t) {
          u32 info;
          while (!((info = __builtin_amdgcn_readfirstlane(ctrl[24 + t])) & ST_VALID)) __builtin_amdgcn_s_sleep(1);
          __builtin_amdgcn_fence(__ATOMIC_ACQUIRE, "workgroup");
          const u32x4 d0 = *(const LAS u32x4*)(lds + L_STG + t * PIECE + lane * 16);
          const u32x4 d1 = *(const LAS u32x4*)(lds + L_STG + t * PIECE + 1024 + lane * 16);
          const u64 due = t0 + (u32)__builtin_amdgcn_readlane(dueo, t);
          while (rt() < due) __builtin_amdgcn_s_sleep(1);
          send(o0 + t * PIECE, CSTRIDE, d0, d1, __ballot(1024 + lane * 16 < ((info >> 16) & 0xfff)));
        }
        ctrl[7] = 1;
        STAMP(2);
        u32x4 fv = *(const LAS u32x4*)(lds + L_CTRL + 96 + (lane & 1) * 16);
        fv = (fv & 0x1fffu) | (call << 13);
        const u32 k = gb / NDC, r = gb - k * NDC;
        char* fa = fws + W_CFLAG + (call & 1) * CFLAG_PAR + ((r * 7 + 6 - fq) * 8 + k) * 128 + (lane & 1) * 16;
        wait_vm<0>();
        if (lane < 14) asm volatile("global_store_dwordx4 %0, %1, off sc0 sc1\n\ts_nop 1" :: "v"(fa), "v"(fv) : "memory");
        STAMP(3);
        sendb = false;
        continue;
      }
      if (sendt && !sendb && now >= duet) {
        STAMP(4);
        send(W_DATA + par + (size_t)((8 + (gt >> 5) * 7) * 32 + (gt & 31)) * UNIT + tt * PIECE, 32 * UNIT, q0, q1, ~0ull);
        wait_vm<0>();
        u32 old = 0;
        if (lane == 0) old = __hip_atomic_fetch_add((u32*)(myws + W_SCNT + gt * 64), 1u, __ATOMIC_RELAXED, __HIP_MEMORY_SCOPE_AGENT);
        if ((__builtin_amdgcn_readfirstlane(old) & 7) == 7 && lane < 14) {
          char* fa = fws + W_READY + ((gt & 31) * 64 + 8 + (gt >> 5) * 7 + 6 - fq) * 128 + (lane & 1) * 16;
          asm volatile("global_store_dwordx4 %0, %1, off sc0 sc1\n\ts_nop 1" :: "v"(fa), "v"(u32x4{call, call, call, call}) : "memory");
        }
        STAMP(9);
        sendt = false;
        continue;
      }
      if (now >= nextp && !allseen) {
        const u32x2 f = *(volatile LAS u32x2*)(lds + L_RDY + lane * 8);
        const u64 na = __ballot(f.x < call), nb = __ballot(f.y < call);
        if ((na | nb) == 0) { allseen = true; continue; }
        const int base = __builtin_ctzll(na | nb);
        const int pp = base + (lane >> 1);
        const u64 em = __ballot((lane >> 1) < PWIN && pp < 64 && ((((lane & 1) ? nb : na) >> (pp & 63)) & 1));
        if (vmcnt_now() < 4)
          asm volatile("s_mov_b32 m0, %2\n\ts_mov_b64 exec, %3\n\tglobal_load_lds_dword %0, %1 sc0 sc1\n\ts_mov_b64 exec, -1"
                       :: "v"(rl), "s"(rbase + base * 128), "s"((u32)(L_RDY + base * 8)), "s"(em) : "memory");
        nextp = now + POLLP;
      }
      if (!sendb && !sendt && (allseen || ctrl[2] != 0)) break;
      __builtin_amdgcn_s_sleep(1);
    }
  } else {
    const int d = wv - 10;
    const u32 dr = sidx / 7, dq = sidx - dr * 7;
    const int nj = dr < NDC ? (NB - dr + NDC - 1) / NDC : 0;
    const u32* cfl = (const u32*)(myws + W_CFLAG + (call & 1) * CFLAG_PAR + sidx * 1024 + d * 16) + (lane & 3);
    LAS char* dl = lds + L_DEC + d * DEC_BYTES;
    const u32 ml = L_DEC + d * DEC_BYTES + 1536;
#pragma unroll 1
    for (int j = 0; j < nj; ++j) {
      const u32 g = dr + j * NDC;
      u32 h;
      bool dead = false;
      const u64 t0 = rt();
      for (u32 it = 1;; ++it) {
        h = __hip_atomic_load(cfl + j * 32, __ATOMIC_RELAXED, __HIP_MEMORY_SCOPE_SYSTEM);
        if (__all((h >> 13) == (call & 0x7ffffu))) break;
        if ((it & 63) == 0 && (__hip_atomic_load(tmo, __ATOMIC_RELAXED, __HIP_MEMORY_SCOPE_AGENT) == call || rt() - t0 > DEADLINE)) {
          if (lane == 0) __hip_atomic_store(tmo, call, __ATOMIC_RELAXED, __HIP_MEMORY_SCOPE_AGENT);
          dead = true;
          break;
        }
        __builtin_amdgcn_s_sleep(DSLEEP);
      }
      if (dead) break;
      const char* src = myws + W_CDATA + (call & 1) * CDATA_PAR + (size_t)(g * 7 + dq) * CSTRIDE + d * 4 * PIECE;
      const u32 pos = 8 + (g >> 5) * 7 + dq;
      char* out = myws + W_DATA + par + (size_t)(pos * 32 + (g & 31)) * UNIT + d * 4 * PIECE;
      glds4(src, lane * 16, ml);
#pragma unroll 1
      for (int i = 0; i < 4; ++i) {
        if (i == 0 || i == 3) wait_vm<2>(); else wait_vm<4>();
        const u32 hi = __builtin_amdgcn_readlane(h, i);
        const LAS char* msg = dl + 1536 + (i & 1) * PIECE;
        u32x4 d0, d1;
        if (hi & ST_RAW) { d0 = *(const LAS u32x4*)(msg + lane * 16); d1 = *(const LAS u32x4*)(msg + 1024 + lane * 16); }
        else if (g & 1) decode_piece<true>(msg, dl + 512, dl, lds + L_LUTD, hi, lane, d0, d1);
        else decode_piece<false>(msg, dl + 512, dl, lds + L_LUTD, hi, lane, d0, d1);
        st2_sys(out + i * PIECE, lane * 16, d0, d1);
        if (i < 2)
          asm volatile("s_mov_b32 m0, %2\n\ts_nop 0\n\t"
                       "global_load_lds_dwordx4 %0, %1 sc0 sc1\n\t"
                       "global_load_lds_dwordx4 %0, %1 offset:1024 sc0 sc1"
                       :: "v"(lane * 16), "s"(src + (i + 2) * PIECE), "s"(ml + (i & 1) * PIECE) : "memory");
      }
      wait_vm<0>();
      if (lane == 0 && lds_ticket(ctrl + 32 + j) == 1)
        __hip_atomic_store((u32*)(myws + W_READY + ((g & 31) * 64 + pos) * 128), call, __ATOMIC_RELAXED, __HIP_MEMORY_SCOPE_SYSTEM);
    }
    STAMP(2);
  }
}

extern "C" void kernel_launch(int rank, int nranks, ncclComm_t comm,
                              hipStream_t stream,
                              void* const* d_in, const int* in_sizes, int n_in,
                              void* d_out, int out_size,
                              const int* in_dtypes, int out_dtype,
                              void* const* d_out_all,
                              void* const* d_ws_all, size_t ws_bytes) {
  if (nranks != NG || (unsigned)rank >= (unsigned)NG || n_in < 4 || in_sizes[0] != MM * KL || in_sizes[1] != KL * NN ||
      out_size != MM * NN || ws_bytes < W_END) {
    fprintf(stderr, "kernel_launch: unexpected problem shape\n");
    return;
  }
  HIP_CHECK(hipFuncSetAttribute((const void*)fused_gemm_ag, hipFuncAttributeMaxDynamicSharedMemorySize, LDS_BYTES));
  Params p;
  for (int i = 0; i < NG; ++i) p.ws[i] = (char*)d_ws_all[i];
  p.x = (const unsigned char*)d_in[0];
  p.w = (const unsigned char*)d_in[1];
  p.sx = (const float*)d_in[2];
  p.sw = (const float*)d_in[3];
  p.out = (float*)d_out;
  p.dbg = nullptr;
  p.rank = rank;
  p.pad = 0;
  hipLaunchKernelGGL(fused_gemm_ag, dim3(256), dim3(768), LDS_BYTES, stream, p);
  HIP_CHECK(hipGetLastError());
}
```

```cpp
#include <hip/hip_runtime.h>
#include <rccl/rccl.h>
#include <cstdio>

#define HIP_CHECK(expr) do { hipError_t _e = (expr); if (_e != hipSuccess) { \
    fprintf(stderr, "HIP error %s at %s:%d\n", hipGetErrorString(_e), __FILE__, __LINE__); \
    return; } } while (0)

#define LAS __attribute__((address_space(3)))
#define GAS __attribute__((address_space(1)))
typedef unsigned u32;
typedef unsigned long long u64;
typedef unsigned u32x4 __attribute__((ext_vector_type(4)));
typedef unsigned u32x2 __attribute__((ext_vector_type(2)));
typedef unsigned u32x8 __attribute__((ext_vector_type(8)));
typedef unsigned u32x16 __attribute__((ext_vector_type(16)));
typedef int i32x8 __attribute__((ext_vector_type(8)));
typedef int i32x4 __attribute__((ext_vector_type(4)));
typedef float f32x16 __attribute__((ext_vector_type(16)));
typedef float f32x4 __attribute__((ext_vector_type(4)));

constexpr int NG = 8;
constexpr int MM = 4096, NN = 4096, KL = 512;
constexpr int UNIT = 16384;
constexpr int PIECE = 2048;
constexpr int NSTEP = 64;
constexpr int NSLOT = 4, SLOT = 2 * UNIT;
constexpr int L_CTRL = 0;
constexpr int L_RDY = L_CTRL + 256;
constexpr int L_LUTC = L_RDY + 1024;
constexpr int L_LUTD = L_LUTC + 256;
constexpr int L_DEC = L_LUTD + 2048;
constexpr int DEC_BYTES = 2 * 2048;
constexpr int L_STG = L_DEC + 2 * DEC_BYTES;
constexpr int L_RING = 32768;
static_assert(L_STG + 8 * PIECE + 512 <= L_RING, "LDS");
constexpr int LDS_BYTES = L_RING + NSLOT * SLOT;
static_assert(LDS_BYTES <= 160 * 1024, "LDS");
#define GH 18
#define NT 24
#define NDC 34
constexpr int NFREE = 256 - 7 * NDC;
constexpr u32 NJ = 512 - 2 * GH, JOFF = (NDC - NJ % NDC) % NDC;
constexpr int CSTRIDE = UNIT + 256;
constexpr size_t W_DATA = 0;
constexpr u32 PSTR = 32 * UNIT;
constexpr size_t DATA_PAR = (size_t)NSTEP * PSTR;
constexpr size_t W_OUT = 2 * DATA_PAR;
constexpr size_t OUT_PAR = (size_t)256 * CSTRIDE;
constexpr size_t W_CFLAG = W_OUT + 2 * OUT_PAR;
constexpr size_t CFLAG_PAR = 256 * 1024;
constexpr size_t W_READY = W_CFLAG + 2 * CFLAG_PAR;
#define NREP 2
constexpr u32 RSTR = 768, REPSTR = 32 * RSTR + 256;
constexpr size_t W_CNT = W_READY + 8 * REPSTR;
constexpr size_t W_TMO = W_CNT + 256 * 128;
constexpr size_t W_SLIP = W_TMO + 256;
constexpr size_t W_TOK = W_SLIP + 32 * 256;
constexpr size_t W_HELLO = W_TOK + 256 * 128;
constexpr size_t W_END = W_HELLO + 8 * 256;
static_assert(W_END < (1ull << 31), "32-bit offsets");
constexpr u64 DEADLINE = 100000000ull;
#define DT_PK 680
#define PACE_T0 525
#define HEAD_T0 85
#define DT_HEAD 945
#define GDEF 48
#define FLAT 90
#define WIN 27
static_assert(WIN < NDC, "token goes round once at most");
#define GRACE 100
#define TGRACE 10
#define ANN_T0 750
#define QUIET 250
#define EPRIO 1
#define TSTART 0
#define LPOLL 0
#define LPACE 88
#define LP0 450
#define DPOLL 2
#define SKCAP 16
#define RAWX 31
#define ANN_DT 19
#define HPRE 1
#define OLEAD 45
#define FPOLL 45
#define SKA 8
#define SKB 0
#define SKC 2
#define LAG0 150
#define LGAIN 0
#define CSLEEP 1
#define PSLEEP 1
#define PGUARD 45
#define LSLEEP 1
#define DSLEEP 1
#define POLLP 40
#define BROT 0
constexpr u32 ST_RAW = 1u << 12, ST_VALID = 1u << 31;

struct Params {
  const unsigned char* x;
  const unsigned char* w;
  const float* sx;
  const float* sw;
  float* out;
  char* own;
  u64* dbg;
  char* peer[7];
  u32 plan[256];
  u32 order[64];
};

#define GPOL " sc0 sc1"
template <int N> __device__ __forceinline__ void wait_vm() { asm volatile("s_waitcnt vmcnt(%0)" :: "i"(N) : "memory"); }
__device__ __forceinline__ u32 vmcnt_now() {
  u32 r;
  asm volatile("s_getreg_b32 %0, hwreg(HW_REG_IB_STS)" : "=s"(r) :: "memory");
  return (r & 15) | ((r >> 18) & 0x30);
}
__device__ __forceinline__ u32 umin32(u32 a, u32 b) { return a < b ? a : b; }
__device__ __forceinline__ u64 rt() { return __builtin_amdgcn_s_memrealtime(); }
#define STR2(x) #x
#define STR(x) STR2(x)
#define NP "\n\t"
#define NH "0"
__device__ __forceinline__ void glds4(const char* base, u32 voff, u32 ldsaddr) {
  asm volatile("s_mov_b32 m0, %2\n\ts_nop " NH "\n\t"
               "global_load_lds_dwordx4 %0, %1" GPOL NP
               "global_load_lds_dwordx4 %0, %1 offset:1024" GPOL NP
               "global_load_lds_dwordx4 %0, %1 offset:2048" GPOL NP
               "global_load_lds_dwordx4 %0, %1 offset:3072" GPOL
               :: "v"(voff), "s"(base), "s"(ldsaddr) : "memory");
}
__device__ __forceinline__ void st2_sys(char* base, u32 voff, u32x4 a, u32x4 b) {
  asm volatile("s_nop 4\n\tglobal_store_dwordx4 %0, %1, %3 sc0 sc1\n\t"
               "global_store_dwordx4 %0, %2, %3 offset:1024 sc0 sc1\n\ts_nop 1"
               :: "v"(voff), "v"(a), "v"(b), "s"(base) : "memory");
}
__device__ __forceinline__ i32x8 ldfrag(const LAS char* p, int hi_off) {
  i32x4 lo = *(const LAS i32x4*)p, hi = *(const LAS i32x4*)(p + hi_off);
  return __builtin_shufflevector(lo, hi, 0, 1, 2, 3, 4, 5, 6, 7);
}
#define MFMA(a, b, c) __builtin_amdgcn_mfma_scale_f32_32x32x64_f8f6f4(a, b, c, 0, 1, 0, 0, 0, 0)
#define SB __builtin_amdgcn_sched_barrier(0)
#define HOLD(v) do {} while (0)
#define PUT(i, v) do {} while (0)
#define STAMP(i) do {} while (0)

__device__ __forceinline__ void tr4(u32 r0, u32 r1, u32 r2, u32 r3, u32& c0, u32& c1, u32& c2, u32& c3) {
  u32 a = __builtin_amdgcn_perm(r1, r0, 0x05010400u);
  u32 b = __builtin_amdgcn_perm(r1, r0, 0x07030602u);
  u32 c = __builtin_amdgcn_perm(r3, r2, 0x05010400u);
  u32 d = __builtin_amdgcn_perm(r3, r2, 0x07030602u);
  c0 = __builtin_amdgcn_perm(c, a, 0x05040100u);
  c1 = __builtin_amdgcn_perm(c, a, 0x07060302u);
  c2 = __builtin_amdgcn_perm(d, b, 0x05040100u);
  c3 = __builtin_amdgcn_perm(d, b, 0x07060302u);
}

#define CFENCE() asm volatile("" ::: "memory")
__device__ __forceinline__ u32 rotr32(u32 x, u32 n) { return __builtin_amdgcn_alignbit(x, x, n); }
__device__ __forceinline__ u32 bfi(u32 m, u32 a, u32 b) { return (a & m) | (b & ~m); }
__device__ __forceinline__ u32 wave_incl_scan(u32 v) {
  v += (u32)__builtin_amdgcn_update_dpp(0, (int)v, 0x111, 0xf, 0xf, true);
  v += (u32)__builtin_amdgcn_update_dpp(0, (int)v, 0x112, 0xf, 0xf, true);
  v += (u32)__builtin_amdgcn_update_dpp(0, (int)v, 0x114, 0xf, 0xf, true);
  v += (u32)__builtin_amdgcn_update_dpp(0, (int)v, 0x118, 0xf, 0xf, true);
  v += (u32)__builtin_amdgcn_update_dpp(0, (int)v, 0x142, 0xa, 0xf, false);
  v += (u32)__builtin_amdgcn_update_dpp(0, (int)v, 0x143, 0xc, 0xf, false);
  return v;
}
__device__ __forceinline__ u32x2 lutd_entry(u32 idx) {
  u32 s = 0, r = 0;
#pragma unroll
  for (int i = 0; i < 4; ++i) { u32 c = (idx >> (2 * i)) & 3; s |= (c == 3 ? r : 4 + c) << (8 * i); r += c == 3; }
  return u32x2{s, r};
}
__device__ __forceinline__ u32x4 lutc_entry(u32 pat) {
  u32 s = 0x0c0c0c0cu, r = 0;
#pragma unroll
  for (int i = 0; i < 4; ++i) if ((pat >> i) & 1) { s = (s & ~(0xffu << (8 * r))) | ((u32)i << (8 * r)); ++r; }
  u32 sw = __builtin_amdgcn_perm(s, s, 0x02030001u);
  return u32x4{s, 2 * r, sw, 4 * r};
}
__device__ __forceinline__ void lds_or_bits(LAS char* base, u32 bitpos, u32 x) {
  u64 y = (u64)x << (bitpos & 31);
  LAS u32* q = (LAS u32*)(base + ((bitpos >> 5) << 2));
  if ((u32)y) __hip_atomic_fetch_or(q, (u32)y, __ATOMIC_RELAXED, __HIP_MEMORY_SCOPE_WORKGROUP);
  if ((u32)(y >> 32)) __hip_atomic_fetch_or(q + 1, (u32)(y >> 32), __ATOMIC_RELAXED, __HIP_MEMORY_SCOPE_WORKGROUP);
}
__device__ __forceinline__ u32 encode_piece(const bool ISW, const u32x4& q0, const u32x4& q1, LAS char* msg, int slot,
                                            volatile LAS u32* alloc, const LAS char* lutc, int lane, bool plain) {
  const u32 MB = ISW ? 2 : 3, EM1 = ISW ? 31 : 15, EM = EM1 * 0x01010101u, FIX = ISW ? 1280 : 1536;
  const u32 NLOW = ISW ? 0x03030303u : 0x0f0f0f0fu, NFIT = ISW ? 0x6c6c6c6cu : 0x78787878u;
  const u32 v[8] = {q0.x, q0.y, q0.z, q0.w, q1.x, q1.y, q1.z, q1.w};
  u32 ex[8], sum = 0;
#pragma unroll
  for (int j = 0; j < 8; ++j) { ex[j] = (v[j] >> MB) & EM; sum = __builtin_amdgcn_sad_u8(ex[j], 0, sum); }
  const u32 tot = __builtin_amdgcn_readlane(wave_incl_scan(sum), 63);
  const u32 c = (tot + 1638) >> 11;
  const u32 kadd = ((EM1 + 1 + 3 - c + EM1 + 1) & EM1) * 0x01010101u;
  constexpr u32 TL = 0x10003534u, TH = 0x3f3f3620u;
  u32 l1[2] = {0, 0}, a2[2] = {0, 0}, c3[2] = {0, 0}, p2[2] = {0, 0}, p3[2] = {0, 0}, bad = 0, a4 = 0, p4 = 0;
#pragma unroll
  for (int w = 0; w < 2; ++w) {
    l1[0] = l1[1]; a2[0] = a2[1]; c3[0] = c3[1]; p2[0] = p2[1]; p3[0] = p3[1];
    u32 l = 0, a = 0, pa = 0, x = 0, px = 0;
#pragma unroll
    for (int k = 0; k < 4; ++k) {
      const u32 r = (ex[k] + kadd) & EM;
      ex[k] = ex[4 + k];
      const u32 t8 = r + 0x78787878u;
      const u32 sel = (t8 & 0x80808080u) | r;
      const u32 cc = __builtin_amdgcn_perm(TH, TL, sel);
      const u32x2 e1 = *(const LAS u32x2*)(lutc + __builtin_amdgcn_udot4(cc & 0x04040404u, 0x20100804u, 0, false));
      const u32x2 e2 = *(const LAS u32x2*)(lutc + 8 + __builtin_amdgcn_udot4(cc & 0x08080808u, 0x10080402u, 0, false));
      const u32 i1 = __builtin_amdgcn_udot4(cc & 0x30303030u, 0x40100401u, 0, false);
      l |= k == 0 ? i1 >> 4 : i1 << (8 * k - 4);
      const u32 c2 = __builtin_amdgcn_udot4(__builtin_amdgcn_perm(cc, cc, e1.x) & 0x03030303u, 0x40100401u, 0, false);
      a |= c2 << pa; pa += e1.y;
      const u32 nib = r & __builtin_amdgcn_perm(NLOW, NLOW, sel) & 0x0f0f0f0fu;
      bad |= t8 & ~(r + NFIT);
      const u32 z = __builtin_amdgcn_perm(nib, nib, e2.x);
      x |= __builtin_amdgcn_perm(0, (z << 12) | z, 0x0c0c0301u) << (px & 31); px += e2.y;
    }
    const u32 lo = x | (x >> 1);
    const u32 g = (x >> 3) & (x >> 2) & lo & 0x11111111u;
    u32 e = (lo | (lo >> 2)) & 0x11111111u & ~g;
    u32 y = (~x & (g | (g << 1))) | e | (e << 1);
    y = (y | (y >> 2)) & 0x0f0f0f0fu;
    while (__any(e != 0)) {
      if (e != 0) { a4 |= ((x >> __builtin_ctz(e)) & 15) << (p4 & 31); p4 += 4; e &= e - 1; }
    }
    l1[1] = l; a2[1] = a; c3[1] = __builtin_amdgcn_perm(0, y | (y >> 4), 0x0c0c0200u); p2[1] = pa; p3[1] = px;
  }
  const bool lbad = (bad & 0x80808080u) != 0 || p3[0] > 32 || p3[1] > 32 || p4 > 28;
  const u32 n = ((p2[0] + p2[1]) >> 1) | ((p3[0] + p3[1]) << 10) | (p4 << 21);
  const u32 inc = wave_incl_scan(n);
  const u32 totn = __builtin_amdgcn_readlane(inc, 63);
  const u32 w2 = ((totn & 0xfff) + 15) >> 4, w3 = (((totn >> 12) & 0x7ff) + 15) >> 4, w4 = ((totn >> 23) + 7) >> 3;
  const bool raw = plain || __any(lbad) || w2 > 64 || w3 > 32 || w2 + w3 + w4 > 127;
  const u32 words = raw ? (PIECE - FIX) / 4 : w2 + w3 + w4;
  u32 toff = 0;
  if (lane == 0) toff = __hip_atomic_fetch_add((LAS u32*)alloc, words, __ATOMIC_RELAXED, __HIP_MEMORY_SCOPE_WORKGROUP);
  toff = __builtin_amdgcn_readfirstlane(toff);
  LAS char* fix = msg + slot * FIX;
  LAS u32* tail = (LAS u32*)(msg + 4 * FIX) + toff;
  if (raw) {
    *(LAS u32x4*)(fix + lane * 16) = q0;
    if (!ISW) { *(LAS u32x2*)(fix + 1024 + lane * 8) = u32x2{q1.x, q1.y}; tail[lane] = q1.z; tail[64 + lane] = q1.w; }
    else { *(LAS u32*)(fix + 1024 + lane * 4) = q1.x; tail[lane] = q1.z; tail[64 + lane] = q1.w; tail[128 + lane] = q1.y; }
    return ST_RAW | (toff << 13);
  }
  if ((u32)lane < words) tail[lane] = 0;
  if ((u32)lane + 64 < words) tail[64 + lane] = 0;
  u32 sm[4];
  if (!ISW) {
#pragma unroll
    for (int k = 0; k < 4; ++k) sm[k] = bfi(0x87878787u, v[2 * k], rotr32(v[2 * k + 1], 28));
    *(LAS u32x2*)(fix + 1024 + lane * 8) = u32x2{sm[2], sm[3]};
  } else {
#pragma unroll
    for (int k = 0; k < 3; ++k) sm[k] = bfi(0x83838383u, v[2 * k], rotr32(v[2 * k + 1], 29));
    sm[0] = bfi(0x60606060u, v[6] << 5, sm[0]);
    sm[1] = bfi(0x60606060u, v[7] << 5, sm[1]);
    sm[2] = bfi(0x60606060u, bfi(0x20202020u, v[6] >> 2, v[7] >> 1), sm[2]);
    *(LAS u32*)(fix + 1024 + lane * 4) = sm[2];
  }
  *(LAS u32x4*)(fix + lane * 16) = u32x4{l1[0], l1[1], sm[0], sm[1]};
  const u32 exc = inc - n;
  const u32 b2 = 2 * (exc & 0xfff), b3 = 2 * ((exc >> 12) & 0x7ff) + 32 * w2, b4 = 4 * (exc >> 23) + 32 * (w2 + w3);
  lds_or_bits((LAS char*)tail, b2, a2[0]);
  lds_or_bits((LAS char*)tail, b2 + p2[0], a2[1]);
  lds_or_bits((LAS char*)tail, b3, c3[0] | (c3[1] << (p3[0] >> 1)));
  lds_or_bits((LAS char*)tail, b4, a4);
  return (c & 31) | (w2 << 5) | (toff << 13) | (w3 << 23);
}
__device__ __forceinline__ u32 lds_ld(u32 a) { return *(const LAS u32*)(size_t)a; }
template <bool ISW>
__device__ __forceinline__ void decode_pair(const u32x4 (&A)[2], const u32x4 (&B)[2], const u32 (&C)[2], LAS char* scr,
                                            const LAS char* lutd, const u32 (&hdr)[2], int lane, u32x4 (&q0)[2], u32x4 (&q1)[2]) {
  constexpr u32 MB = ISW ? 2 : 3, EM1 = ISW ? 31 : 15, EM = EM1 * 0x01010101u;
  u32 sel[2][14], cn[2][14], ad[2][14], r0[2][14], r1[2][14], x[2][14], t1[2], t2[2], t3[2];
#pragma unroll
  for (int m = 0; m < 2; ++m) {
    const u32 cw[4] = {B[m].z, A[m].x, A[m].y, B[m].w};
#pragma unroll
    for (int i = 0; i < 14; ++i) {
      u32 a;
      if ((i & 3) == 0) asm("v_lshlrev_b32_sdwa %0, %1, %2 dst_sel:DWORD dst_unused:UNUSED_PAD src0_sel:DWORD src1_sel:BYTE_0" : "=v"(a) : "v"(3), "v"(cw[i >> 2]));
      else a = ((cw[i >> 2] >> (8 * (i & 3))) & 0xff) << 3;
      const u32x2 e = *(const LAS u32x2*)(lutd + a);
      sel[m][i] = e.x; cn[m][i] = e.y;
    }
  }
  SB;
#pragma unroll
  for (int m = 0; m < 2; ++m) {
    const u32 b4 = (((hdr[m] & 31) + EM1 + 1 - 3) & EM1) * 0x01010101u;
    t1[m] = ((b4 + 0x00040302u) & EM) << MB; t2[m] = ((b4 + 0x00050100u) & EM) << MB; t3[m] = ((b4 + 0x001d1e1fu) & EM) << MB;
    u32 lo = C[m] & 0x0f0f0f0fu, hi = (C[m] >> 4) & 0x0f0f0f0fu;
    if (ISW) {
      const u32 gl = ~(lo + 0x7c7c7c7cu) & 0x80808080u, gh = ~(hi + 0x7c7c7c7cu) & 0x80808080u;
      lo ^= (gl >> 3) | (gl >> 5); hi ^= (gh >> 3) | (gh >> 5);
    }
    const u32 bb = ISW ? b4 + 0x10101010u : b4;
    lo = ((lo + bb) & EM) << MB; hi = ((hi + bb) & EM) << MB;
    *(LAS u32x2*)(scr + m * 2048 + lane * 8) = u32x2{__builtin_amdgcn_perm(hi, lo, 0x05010400u), __builtin_amdgcn_perm(hi, lo, 0x07030602u)};
  }
  CFENCE();
  SB;
#pragma unroll
  for (int m = 0; m < 2; ++m) {
    const u32* c = cn[m];
    const u32 n1 = c[4] + c[5] + c[6] + c[7] + c[8] + c[9] + c[10] + c[11];
    const u32 n = n1 | ((c[0] + c[1] + c[2] + c[3]) << 12) | ((c[12] + c[13]) << 23);
    const u32 exc = wave_incl_scan(n) - n;
    const u32 sb = (u32)(size_t)scr + m * 2048;
    ad[m][0] = sb + 512 + ((exc >> 12) & 0x3ff); ad[m][4] = sb + 1024 + (exc & 0x7ff); ad[m][12] = sb + (exc >> 23);
#pragma unroll
    for (int i = 1; i < 14; ++i) if (i != 4 && i != 12) ad[m][i] = ad[m][i - 1] + c[i - 1];
#pragma unroll
    for (int i = 12; i < 14; ++i) { r0[m][i] = lds_ld(ad[m][i] & ~3u); r1[m][i] = lds_ld((ad[m][i] & ~3u) + 4); }
  }
  SB;
#pragma unroll
  for (int m = 0; m < 2; ++m) {
#pragma unroll
    for (int i = 12; i < 14; ++i) x[m][i] = __builtin_amdgcn_perm(t3[m], __builtin_amdgcn_alignbyte(r1[m][i], r0[m][i], ad[m][i]), sel[m][i]);
    *(LAS u32x2*)(scr + m * 2048 + 512 + lane * 8) = u32x2{x[m][12], x[m][13]};
    CFENCE();
#pragma unroll
    for (int i = 0; i < 4; ++i) { r0[m][i] = lds_ld(ad[m][i] & ~3u); r1[m][i] = lds_ld((ad[m][i] & ~3u) + 4); }
    CFENCE();
  }
  SB;
#pragma unroll
  for (int m = 0; m < 2; ++m) {
#pragma unroll
    for (int i = 0; i < 4; ++i) x[m][i] = __builtin_amdgcn_perm(t2[m], __builtin_amdgcn_alignbyte(r1[m][i], r0[m][i], ad[m][i]), sel[m][i]);
    *(LAS u32x4*)(scr + m * 2048 + 1024 + lane * 16) = u32x4{x[m][0], x[m][1], x[m][2], x[m][3]};
    CFENCE();
#pragma unroll
    for (int i = 4; i < 12; ++i) { r0[m][i] = lds_ld(ad[m][i] & ~3u); r1[m][i] = lds_ld((ad[m][i] & ~3u) + 4); }
    CFENCE();
  }
  SB;
#pragma unroll
  for (int m = 0; m < 2; ++m) {
#pragma unroll
    for (int i = 4; i < 12; ++i) x[m][i] = __builtin_amdgcn_perm(t1[m], __builtin_amdgcn_alignbyte(r1[m][i], r0[m][i], ad[m][i]), sel[m][i]);
    const u32 s0 = A[m].z, s1 = A[m].w, s2 = B[m].x, s3 = B[m].y;
    if (!ISW) {
      constexpr u32 S = 0x87878787u;
      q0[m] = u32x4{(s0 & S) | x[m][4], (rotr32(s0, 4) & S) | x[m][5], (s1 & S) | x[m][6], (rotr32(s1, 4) & S) | x[m][7]};
      q1[m] = u32x4{(s2 & S) | x[m][8], (rotr32(s2, 4) & S) | x[m][9], (s3 & S) | x[m][10], (rotr32(s3, 4) & S) | x[m][11]};
    } else {
      constexpr u32 S = 0x83838383u;
      q0[m] = u32x4{(s0 & S) | x[m][4], (rotr32(s0, 3) & S) | x[m][5], (s1 & S) | x[m][6], (rotr32(s1, 3) & S) | x[m][7]};
      q1[m] = u32x4{(s2 & S) | x[m][8], (rotr32(s2, 3) & S) | x[m][9],
                    (bfi(0x03030303u, s0 >> 5, s2 << 2) & S) | x[m][10], (bfi(0x03030303u, s1 >> 5, s2 << 1) & S) | x[m][11]};
    }
  }
  CFENCE();
}

__device__ __forceinline__ void fetch_piece(const unsigned char* px, const unsigned char* pw, int hc, int m, int t, int lane, u32x4& q0, u32x4& q1) {
  if (t < 4) {
    const GAS unsigned char* src = (const GAS unsigned char*)px + (size_t)(m * 256 + t * 64 + lane) * KL + hc * 32;
    q0 = *(const GAS u32x4*)src; q1 = *(const GAS u32x4*)(src + 16);
  } else {
    const GAS unsigned char* src = (const GAS unsigned char*)pw + (size_t)(hc * 32 + (t & 1) * 16 + (lane >> 5) * 8) * NN + m * 256 + ((t >> 1) & 1) * 128 + (lane & 31) * 4;
#pragma unroll
    for (int k = 0; k < 4; ++k) { q0[k] = *(const GAS u32*)(src + (size_t)k * NN); q1[k] = *(const GAS u32*)(src + (size_t)(k + 4) * NN); }
  }
}
__device__ __forceinline__ void arrange_piece(int t, u32x4& q0, u32x4& q1) {
  if (t >= 4) {
    u32 a[4], b[4];
    tr4(q0.x, q0.y, q0.z, q0.w, a[0], a[1], a[2], a[3]);
    tr4(q1.x, q1.y, q1.z, q1.w, b[0], b[1], b[2], b[3]);
    auto s0 = __builtin_amdgcn_permlane32_swap(a[0], a[1], false, false);
    auto s1 = __builtin_amdgcn_permlane32_swap(b[0], b[1], false, false);
    auto s2 = __builtin_amdgcn_permlane32_swap(a[2], a[3], false, false);
    auto s3 = __builtin_amdgcn_permlane32_swap(b[2], b[3], false, false);
    q0 = u32x4{s0[0], s1[0], s0[1], s1[1]};
    q1 = u32x4{s2[0], s3[0], s2[1], s3[1]};
  }
}
__device__ __forceinline__ u32 data_off(u32 v, u32 unit) { return (v >> 1) * PSTR + unit * UNIT + (v & 1) * (UNIT / 2); }
__device__ __forceinline__ u32 ready_off(u32 v, u32 unit) { return unit * RSTR + v * 4; }
__device__ __forceinline__ u32 lds_ticket(volatile LAS u32* w, int lane) {
  u32 t = 0;
  if (lane == 0) t = __hip_atomic_fetch_add((LAS u32*)w, 1u, __ATOMIC_RELAXED, __HIP_MEMORY_SCOPE_WORKGROUP);
  return __builtin_amdgcn_readfirstlane(t);
}

__global__ void __launch_bounds__(768, 3) fused_gemm_ag(Params p) {
  LAS char* const lds = (LAS char*)(size_t)0;
  volatile LAS u32* ctrl = (volatile LAS u32*)(lds + L_CTRL);
  const u64 tstart = rt();
  const int tid = threadIdx.x, lane = tid & 63, wv = __builtin_amdgcn_readfirstlane(tid >> 6);
  const int bid = blockIdx.x;
  u32 plan = p.plan[bid];
  const unsigned char *px = p.x, *pw = p.w;
  char* myws = p.own;
  if ((u32)(tid - 512) < 80) *(LAS u32x4*)(lds + (tid < 576 ? L_RDY - 512 * 16 : L_CTRL - 576 * 16) + tid * 16) = u32x4{0, tid == 576 + 13 ? ~0u : 0, 0, 0};
  asm volatile("s_waitcnt lgkmcnt(0)" : "+s"(plan), "+s"(px), "+s"(pw), "+s"(myws) :: "memory");
  if (wv == 9)
    asm volatile("s_mov_b32 m0, %2\n\ts_mov_b64 exec, 1\n\ts_nop 2\n\tglobal_load_lds_dword %0, %1 sc1\n\ts_mov_b64 exec, -1"
                 :: "v"(0), "s"(myws + W_CNT + bid * 128), "s"((u32)(L_CTRL + 53 * 4)) : "memory");
  const int sidx = plan & 255, ti = (plan >> 8) & 15, tj = (plan >> 12) & 15;
  const int gb = plan >> 24, bh = (plan >> 20) & 15, bm = (plan >> 16) & 15;
  u64 h1 = 0, h5 = 0, h6 = 0, h7 = 0; (void)h1; (void)h5; (void)h6; (void)h7;

  HOLD(h1);
  u32x4 q0, q1;
  if (wv < 8) fetch_piece(px, pw, bh, bm, wv, lane, q0, q1);
  asm volatile("s_barrier" ::: "memory");
  auto get_call = [&]() -> u32 {
    u32 c;
    while ((c = __builtin_amdgcn_readfirstlane(ctrl[53])) == ~0u) __builtin_amdgcn_s_sleep(1);
    return c + 1;
  };
  const bool head = gb < GH && wv < 8;
  if (__builtin_expect(head, 1)) {
    char* pe[7];
#pragma unroll
    for (int q = 0; q < 7; ++q) pe[q] = p.peer[q] + data_off(16 + 7 * bh + 6 - q, 2 * bm + (wv >> 2));
    asm volatile("" : "+s"(pe[0]), "+s"(pe[1]), "+s"(pe[2]), "+s"(pe[3]), "+s"(pe[4]), "+s"(pe[5]), "+s"(pe[6]));
    const u32 o = (u32)(W_DATA + (get_call() & 1) * DATA_PAR) + (wv & 3) * PIECE + lane * 16;
    arrange_piece(wv, q0, q1);
    while ((u32)rt() - (u32)tstart < HEAD_T0 + ((wv * GH + gb) * DT_HEAD >> 8)) __builtin_amdgcn_s_sleep(1);
#define HS(n) "global_store_dwordx4 %0, %1, %" #n " sc0 sc1\n\tglobal_store_dwordx4 %0, %2, %" #n " offset:1024 sc0 sc1\n\t"
    asm volatile("s_nop 4\n\t" HS(3) HS(4) HS(5) HS(6) HS(7) HS(8) HS(9) "s_nop 1"
                 :: "v"(o), "v"(q0), "v"(q1), "s"(pe[0]), "s"(pe[1]), "s"(pe[2]), "s"(pe[3]), "s"(pe[4]), "s"(pe[5]), "s"(pe[6]) : "memory");
  }
  if (wv >= 8) __builtin_amdgcn_s_setprio(3);
  if (wv == 10) {
    if (lane < 16) *(LAS u32x4*)(lds + L_LUTC + lane * 16) = lutc_entry(lane);
    asm volatile("s_waitcnt lgkmcnt(0)" ::: "memory");
    ctrl[52] = 1;
  }
  if (wv == 11) {
#pragma unroll
    for (int i = 0; i < 4; ++i) *(LAS u32x2*)(lds + L_LUTD + (i * 64 + lane) * 8) = lutd_entry(i * 64 + lane);
    asm volatile("s_waitcnt lgkmcnt(0)" ::: "memory");
    ctrl[51] = 1;
  }
  if (wv == 8) ctrl[4] = __builtin_bit_cast(u32, p.sx[0] * p.sw[0]);
  if (wv < 8 && !head) {
    arrange_piece(wv, q0, q1);
    u32 hdr;
    if (EPRIO) __builtin_amdgcn_s_setprio(3);
    while (ctrl[52] == 0) __builtin_amdgcn_s_sleep(1);
    hdr = encode_piece(wv >= 4, q0, q1, lds + L_STG + (wv & 4) * PIECE, wv & 3, ctrl + 54 + (wv >> 2), lds + L_LUTC, lane, gb >= 256 - NT);
    asm volatile("s_waitcnt lgkmcnt(0)" ::: "memory");
    ctrl[24 + wv] = ST_VALID | hdr;
    __builtin_amdgcn_s_setprio(0);
  }
  const u32 call = get_call();
  auto publish = [&](u32 v, u32 unit) {
    if (lane < NREP)
      __hip_atomic_store((GAS u32*)(myws + W_READY + lane * REPSTR + ready_off(v, unit)), call, __ATOMIC_RELAXED, __HIP_MEMORY_SCOPE_SYSTEM);
  };
  auto move_origin = [&](u32 link, u64 t) {
    if (lane < 4) __hip_atomic_fetch_max((GAS u64*)(myws + W_SLIP + (lane * 8 + link) * 256), t, __ATOMIC_RELAXED, __HIP_MEMORY_SCOPE_AGENT);
  };
  GAS u32* const tmo = (GAS u32*)(myws + W_TMO);
  const size_t par = (call & 1) * DATA_PAR;
  const u32 ctag = call % 0xfffu + 1;
  const u32 di = sidx >= NFREE ? sidx - NFREE : 0, dr = di / 7, dq = di - dr * 7;
  const int j0 = dr < JOFF ? 1 : 0, j1 = sidx >= NFREE ? (NJ + JOFF) / NDC : 0;

  const char* ra = myws + W_READY + (tj % NREP) * REPSTR + 2 * ti * RSTR;
  const char* rb = myws + W_READY + (ti % NREP) * REPSTR + (2 * tj + 1) * RSTR;
  u32 ba = 0, bb = 0;
  auto poll = [&]() {
    const u32 wa = umin32(ba, 112), wb = umin32(bb, 112);
    u32x16 fa, fb;
    asm volatile("s_load_dwordx16 %0, %2, %4 glc\n\ts_load_dwordx16 %1, %3, %5 glc\n\ts_waitcnt lgkmcnt(0)"
                 : "=&s"(fa), "=&s"(fb) : "s"(ra), "s"(rb), "s"(wa * 4), "s"(wb * 4) : "memory");
    u32 v = 0;
#define WL4(x, i, l) asm("v_writelane_b32 %0, %1, " #l "\n\tv_writelane_b32 %0, %2, " #l "+1\n\tv_writelane_b32 %0, %3, " #l "+2\n\tv_writelane_b32 %0, %4, " #l "+3" \
                       : "+v"(v) : "s"(x[i]), "s"(x[i + 1]), "s"(x[i + 2]), "s"(x[i + 3]))
    WL4(fa, 0, 0); WL4(fa, 4, 4); WL4(fa, 8, 8); WL4(fa, 12, 12);
    WL4(fb, 0, 16); WL4(fb, 4, 20); WL4(fb, 8, 24); WL4(fb, 12, 28);
    const u32 sl = (lane < 16 ? wa : wb) + (lane & 15);
    if (lane < 32)
      __hip_atomic_fetch_max((LAS u32*)(lds + L_RDY + (sl >> 1) * 16 + (lane >> 4) * 8 + (sl & 1) * 4), v, __ATOMIC_RELAXED, __HIP_MEMORY_SCOPE_WORKGROUP);
    const u32 m = ~(u32)__ballot(v >= call);
    ba = wa + __builtin_ctz(m | 0x10000u); bb = wb + __builtin_ctz((m >> 16) | 0x10000u);
  };

  if (wv < 8) {
    {
      st2_sys(myws + W_DATA + par + data_off(bh, 2 * bm + (wv >> 2)) + (wv & 3) * PIECE, lane * 16, q0, q1);
      wait_vm<0>();
      if (head) ctrl[24 + wv] = ST_VALID;
      if (lds_ticket(ctrl + 48 + (wv >> 2), lane) == 3) publish(bh, 2 * bm + (wv >> 2));
    }

    PUT(0, tstart); PUT(1, h1); PUT(5, h5); PUT(6, h6);
    const int wm = wv & 3, wn = wv >> 2;
    char* ob = (char*)(p.out + (size_t)(ti * 256 + wm * 64) * NN + tj * 256 + wn * 128);
    {
      u32 oo = (4 * (lane >> 5) * NN + 4 * (lane & 31)) * 4;
      u32 va = (lane >> 5) * 8192 + wm * 2048 + (lane & 31) * 16;
      u32 vb = UNIT + (lane >> 5) * 8192 + wn * 4096 + (lane & 31) * 16;
      u32 vc = L_CTRL + wv * 4, vt, ss, sk;
#define FA0 "v[136:143]"
#define FA1 "v[144:151]"
#define FB0 "v[152:159]"
#define FB1 "v[160:167]"
#define MM(c, a, b) "v_mfma_f32_32x32x64_f8f6f4 " c ", " a ", " b ", " c " blgp:1\n\t"
#define RD(lo, hi, ad, o0, o1) "ds_read_b128 " lo ", " ad " offset:32768+" #o0 "\n\tds_read_b128 " hi ", " ad " offset:32768+" #o1 "\n\t"
#define SPIN "1:\n\t" \
        "ds_read_b32 %[vt], %[vc] offset:64\n\t" \
        "s_waitcnt lgkmcnt(0)\n\t" \
        "v_readfirstlane_b32 %[sk], %[vt]\n\t" \
        "s_cmp_gt_u32 %[sk], %[ss]\n\t" \
        "s_cbranch_scc1 2f\n\t" \
        "s_sleep " STR(CSLEEP) "\n\t" \
        "s_branch 1b\n\t" \
        "2:\n\t"
#define MUL4(d, k) "v_mul_f32 v" #d ", v[8+" #k "], %[vt]\n\tv_mul_f32 v[" #d "+1], v[24+" #k "], %[vt]\n\t" \
                   "v_mul_f32 v[" #d "+2], v[40+" #k "], %[vt]\n\tv_mul_f32 v[" #d "+3], v[56+" #k "], %[vt]\n\t"
      asm volatile(
        "s_mov_b32 %[ss], 0\n\t"
        "3:\n\t"
        "s_set_gpr_idx_on %[ss], gpr_idx(DST)\n\t"
        "v_mov_b32 v8, 0\n\tv_mov_b32 v9, 0\n\tv_mov_b32 v10, 0\n\tv_mov_b32 v11, 0\n\t"
        "v_mov_b32 v12, 0\n\tv_mov_b32 v13, 0\n\tv_mov_b32 v14, 0\n\tv_mov_b32 v15, 0\n\t"
        "s_set_gpr_idx_off\n\t"
        "s_add_u32 %[ss], %[ss], 8\n\t"
        "s_cmp_lt_u32 %[ss], 128\n\t"
        "s_cbranch_scc1 3b\n\t"
        "s_mov_b32 %[ss], 0\n\t"
        SPIN
        RD("v[136:139]", "v[140:143]", "%[va]", 0, 1024)
        RD("v[152:155]", "v[156:159]", "%[vb]", 0, 2048)
        RD("v[144:147]", "v[148:151]", "%[va]", 512, 1536)
        RD("v[160:163]", "v[164:167]", "%[vb]", 512, 2560)
        "5:\n\t"
        "s_waitcnt lgkmcnt(4)\n\t"
        MM("v[8:23]", FA0, FB0)
        "s_waitcnt lgkmcnt(2)\n\t"
        MM("v[72:87]", FA1, FB0)
        RD("v[152:155]", "v[156:159]", "%[vb]", 1024, 3072)
        "s_waitcnt lgkmcnt(2)\n\t"
        MM("v[24:39]", FA0, FB1)
        MM("v[88:103]", FA1, FB1)
        RD("v[160:163]", "v[164:167]", "%[vb]", 1536, 3584)
        "ds_read_b32 %[vt], %[vc] offset:64\n\t"
        "v_add_u32 %[va], 0x8000, %[va]\n\t"
        "v_add_u32 %[vb], 0x8000, %[vb]\n\t"
        "s_add_u32 %[ss], %[ss], 1\n\t"
        "s_waitcnt lgkmcnt(3)\n\t"
        MM("v[40:55]", FA0, FB0)
        "v_and_b32 %[va], 0x1ffff, %[va]\n\t"
        "v_and_b32 %[vb], 0x1ffff, %[vb]\n\t"
        MM("v[104:119]", FA1, FB0)
        "s_waitcnt lgkmcnt(0)\n\t"
        "v_readfirstlane_b32 %[sk], %[vt]\n\t"
        "v_mov_b32 %[vt], %[ss]\n\t"
        "ds_write_b32 %[vc], %[vt] offset:32\n\t"
        "s_cmp_eq_u32 %[ss], 64\n\t"
        "s_cbranch_scc1 6f\n\t"
        "s_cmp_gt_u32 %[sk], %[ss]\n\t"
        "s_cbranch_scc1 2f\n\t"
        SPIN
        RD("v[152:155]", "v[156:159]", "%[vb]", 0, 2048)
        MM("v[56:71]", FA0, FB1)
        RD("v[136:139]", "v[140:143]", "%[va]", 0, 1024)
        MM("v[120:135]", FA1, FB1)
        RD("v[144:147]", "v[148:151]", "%[va]", 512, 1536)
        RD("v[160:163]", "v[164:167]", "%[vb]", 512, 2560)
        "s_branch 5b\n\t"
        "6:\n\t"
        MM("v[56:71]", FA0, FB1)
        MM("v[120:135]", FA1, FB1)
        "v_add_u32 %[va], 0x4000, %[oo]\n\t"
        "v_add_u32 %[vb], 0x8000, %[oo]\n\t"
        "v_add_u32 %[vc], 0xc000, %[oo]\n\t"
        "v_mov_b32 %[vt], 0\n\t"
        "7:\n\t"
        "ds_read_b64 v[152:153], %[vt] offset:16\n\t"
        "s_waitcnt lgkmcnt(0)\n\t"
        "v_readfirstlane_b32 %[sk], v153\n\t"
        "s_cmp_lg_u32 %[sk], 0\n\t"
        "s_cbranch_scc1 7b\n\t"
        "v_mov_b32 %[vt], v152\n\t"
        "s_mov_b32 %[ss], 0\n\t"
        "s_nop 15\n\t"
        "8:\n\t"
        "s_set_gpr_idx_on %[ss], gpr_idx(SRC0)\n\t"
        MUL4(136, 0) MUL4(140, 1) MUL4(144, 2) MUL4(148, 3)
        "s_set_gpr_idx_off\n\t"
        "global_store_dwordx4 %[oo], v[136:139], %[ob] sc1\n\t"
        "global_store_dwordx4 %[va], v[140:143], %[ob] sc1\n\t"
        "global_store_dwordx4 %[vb], v[144:147], %[ob] sc1\n\t"
        "global_store_dwordx4 %[vc], v[148:151], %[ob] sc1\n\t"
        "v_add_u32 %[oo], 0x20000, %[oo]\n\t"
        "v_add_u32 %[va], 0x20000, %[va]\n\t"
        "v_add_u32 %[vb], 0x20000, %[vb]\n\t"
        "v_add_u32 %[vc], 0x20000, %[vc]\n\t"
        "s_add_u32 %[ss], %[ss], 4\n\t"
        "s_cmp_eq_u32 %[ss], 16\n\t"
        "s_cselect_b32 %[ss], 64, %[ss]\n\t"
        "s_cmp_lt_u32 %[ss], 80\n\t"
        "s_cbranch_scc1 8b"
        : [va] "+v"(va), [vb] "+v"(vb), [vc] "+v"(vc), [oo] "+v"(oo), [vt] "=&v"(vt), [ss] "=&s"(ss), [sk] "=&s"(sk)
        : [ob] "s"(ob)
        : "memory", "scc",
          "v8","v9","v10","v11","v12","v13","v14","v15","v16","v17","v18","v19","v20","v21","v22","v23","v24","v25","v26","v27",
          "v28","v29","v30","v31","v32","v33","v34","v35","v36","v37","v38","v39","v40","v41","v42","v43","v44","v45","v46","v47",
          "v48","v49","v50","v51","v52","v53","v54","v55","v56","v57","v58","v59","v60","v61","v62","v63","v64","v65","v66","v67",
          "v68","v69","v70","v71","v72","v73","v74","v75","v76","v77","v78","v79","v80","v81","v82","v83","v84","v85","v86","v87",
          "v88","v89","v90","v91","v92","v93","v94","v95","v96","v97","v98","v99","v100","v101","v102","v103","v104","v105","v106","v107",
          "v108","v109","v110","v111","v112","v113","v114","v115","v116","v117","v118","v119","v120","v121","v122","v123","v124","v125","v126","v127",
          "v128","v129","v130","v131","v132","v133","v134","v135","v136","v137","v138","v139","v140","v141","v142","v143","v144","v145","v146","v147",
          "v148","v149","v150","v151","v152","v153","v154","v155","v156","v157","v158","v159","v160","v161","v162","v163","v164","v165","v166","v167");
    }
    STAMP(3);
  } else if (wv == 8) {
    while ((u32)rt() - (u32)tstart < QUIET) __builtin_amdgcn_s_sleep(8);
    const char* srcA = myws + W_DATA + par + (size_t)(2 * ti) * UNIT;
    const char* srcB = myws + W_DATA + par + (size_t)(2 * tj + 1) * UNIT;
    u64 todo0 = ~0ull, todo1 = ~0ull, av0 = 0, av1 = 0;
    int pub = 0;
    bool dead = false;
    volatile LAS u32x4* const mir = (volatile LAS u32x4*)(lds + L_RDY + lane * 16);
    volatile LAS u32* const frw = ctrl + 8 + (lane & 7);
    u32x4 f = {0, 0, 0, 0};
    int fr = 0;
    for (int s = 0; s < NSTEP; ++s) {
      av0 = __ballot(umin32(f.x, f.z) >= call) & todo0; av1 = __ballot(umin32(f.y, f.w) >= call) & todo1;
      if (__builtin_popcountll(av0) + __builtin_popcountll(av1) < 2 && !dead) {
        const u64 t0 = rt();
        for (int it = 1;; ++it) {
          f = *mir; fr = *frw;
          av0 = __ballot(umin32(f.x, f.z) >= call) & todo0; av1 = __ballot(umin32(f.y, f.w) >= call) & todo1;
          if (__builtin_popcountll(av0) + __builtin_popcountll(av1) >= 2) break;
          if (pub < s && vmcnt_now() == 0) { wait_vm<0>(); pub = s; ctrl[16 + (lane & 7)] = pub; }
          if ((it & 255) == 0 && (__hip_atomic_load(tmo, __ATOMIC_RELAXED, __HIP_MEMORY_SCOPE_AGENT) == call || rt() - t0 > DEADLINE)) {
            if (lane == 0) __hip_atomic_store(tmo, call, __ATOMIC_RELAXED, __HIP_MEMORY_SCOPE_AGENT);
            dead = true; ctrl[2] = 1; ctrl[4] = 0x7fc00000u;
            break;
          }
          if (LPOLL && pub == s && s >= LPOLL) {
            ba = bb = 2 * __builtin_ctzll(todo0 | todo1);
            poll();
          } else __builtin_amdgcn_s_sleep(LSLEEP);
        }
      }
      while (LPACE && s < 8 && (u32)rt() - (u32)tstart < LP0 + s * LPACE + (sidx & 15) * (LPACE / 16)) {
        if (pub < s && vmcnt_now() == 0) { wait_vm<0>(); pub = s; ctrl[16 + (lane & 7)] = pub; }
        __builtin_amdgcn_s_sleep(2);
      }
      const int need = s - NSLOT + 1;
      while (!__all(fr >= need)) {
        if (pub < s && vmcnt_now() == 0) { wait_vm<0>(); pub = s; ctrl[16 + (lane & 7)] = pub; }
        fr = *frw;
      }
      if (dead) { wait_vm<0>(); pub = s; ctrl[16 + (lane & 7)] = pub; continue; }
      u32 go[2];
#pragma unroll
      for (int h = 0; h < 2; ++h) {
        const int pos = __builtin_ctzll(av0 | av1);
        const u64 bit = 1ull << pos;
        const bool even = av0 & bit;
        go[h] = pos * PSTR + (even ? 0 : UNIT / 2);
        if (even) { av0 &= ~bit; todo0 &= ~bit; } else { av1 &= ~bit; todo1 &= ~bit; }
      }
      const u32 l = L_RING + (s & (NSLOT - 1)) * SLOT;
#pragma unroll
      for (int j = 0; j < 4; ++j) glds4(srcA + go[j >> 1] + (j & 1) * 4096, lane * 16, l + j * 4096);
      if (pub < s && vmcnt_now() <= 16) { wait_vm<16>(); pub = s; ctrl[16 + (lane & 7)] = pub; }
      glds4(srcB + go[0], lane * 16, l + UNIT); glds4(srcB + go[0] + 4096, lane * 16, l + UNIT + 4096);
      f = *mir; fr = *frw;
      glds4(srcB + go[1], lane * 16, l + UNIT + 2 * 4096);
      if (pub < s) { wait_vm<28>(); pub = s; ctrl[16 + (lane & 7)] = pub; }
      glds4(srcB + go[1] + 4096, lane * 16, l + UNIT + 3 * 4096);
    }
    STAMP(2);
    if (__hip_atomic_load(tmo, __ATOMIC_RELAXED, __HIP_MEMORY_SCOPE_AGENT) == call) { ctrl[2] = 1; ctrl[4] = 0x7fc00000u; }
    wait_vm<0>();
    asm volatile("s_waitcnt lgkmcnt(0)" ::: "memory");
    ctrl[16 + (lane & 7)] = NSTEP;
    STAMP(3);
  } else if (wv == 9) {
    const u64 t0 = tstart;
    const char* const org = myws + W_SLIP + ((dr & 3) * 8 + dq) * 256;
    if (j1) ctrl[5] = 1;
    if (lane == 0) __hip_atomic_store((GAS u32*)(myws + W_CNT + bid * 128), call, __ATOMIC_RELAXED, __HIP_MEMORY_SCOPE_AGENT);
    auto trel = [&]() -> u32 { return (u32)rt() - (u32)t0; };
    u32 nextp = 0;
#define allseen ((ba & bb) == 128)
    const int fq = (lane & 15) < 14 ? (lane & 15) >> 1 : 0;
    char* fa = p.peer[fq];
    u32x4 fv;
    u32 tann = ~0u;
    auto announce = [&](u32 now) {
      if (now < tann) return;
      tann = ~0u;
      if ((lane & 15) < 14 && lane < 32) asm volatile("global_store_dwordx4 %0, %1, off sc0 sc1\n\ts_nop 1" :: "v"(fa), "v"(fv) : "memory");
    };
    auto idle_until = [&](u32 t) {
      u32 now;
      while ((now = trel()) < t) {
        announce(now);
        if (now >= nextp && now + PGUARD < t && !allseen) { poll(); nextp = now + POLLP; }
        __builtin_amdgcn_s_sleep(PSLEEP);
      }
    };
    if (gb < GH) {
      const u32 rep = (lane & 1) + 2 * (lane >> 4);
#pragma unroll 1
      for (int k = 0; k < 2; ++k) {
        while (!__all(ctrl[24 + 4 * k + (lane & 3)] & ST_VALID)) __builtin_amdgcn_s_sleep(1);
        if ((lane & 15) < 14 && rep < NREP)
          __hip_atomic_store((GAS u32*)(fa + W_READY + rep * REPSTR + ready_off(16 + 7 * bh + 6 - fq, 2 * bm + k)), call, __ATOMIC_RELAXED, __HIP_MEMORY_SCOPE_SYSTEM);
      }
    } else {
#pragma unroll 1
      for (int k = 0; k < 2; ++k) {
        while (!__all(ctrl[24 + 4 * k + (lane & 3)] & ST_VALID)) __builtin_amdgcn_s_sleep(2);
        __builtin_amdgcn_fence(__ATOMIC_ACQUIRE, "workgroup");
        char* o = myws + W_OUT + (call & 1) * OUT_PAR + gb * CSTRIDE + k * 4 * PIECE;
#pragma unroll
        for (int t = 0; t < 4; ++t)
          st2_sys(o, t * PIECE + lane * 16, *(const LAS u32x4*)(lds + L_STG + (4 * k + t) * PIECE + lane * 16),
                  *(const LAS u32x4*)(lds + L_STG + (4 * k + t) * PIECE + 1024 + lane * 16));
      }
      const u32 k = (lane >> 4) & 1, n = 2 * (gb - GH) + JOFF, jk = n / NDC, jr = n - jk * NDC + k;
      fa += W_CFLAG + (call & 1) * CFLAG_PAR + ((jr * 7 + 6 - fq) * 16 + jk) * 64 + (lane & 1) * 16;
      const u32x2 h = *(volatile LAS u32x2*)(lds + L_CTRL + 96 + k * 16 + (lane & 1) * 8);
      fv = u32x4{h.x & 0xfffffu, ((h.x >> 20) & 0x1ffu) | (ctrl[54 + k] << 9), h.y & 0xfffffu, (h.y >> 20) & 0x1ffu} | (ctag << 20);
      HOLD(h7);
      wait_vm<0>();
      tann = gb < GDEF ? 0 : ANN_T0 + (gb - GDEF) * ANN_DT;
      announce(trel());
    }
    STAMP(2);
    {
      const char* cfl = myws + W_CFLAG + (call & 1) * CFLAG_PAR + di * 1024;
      const char* src0 = p.peer[dq < 7 ? dq : 0] + W_OUT + (call & 1) * OUT_PAR;
      u32 base = 0;
      u32 words = ~0u, hw = 0;
      auto headers_of = [&](int j) {
        u32x8 hv;
        asm volatile("s_load_dwordx8 %0, %1, %2 glc\n\ts_waitcnt lgkmcnt(0)" : "=&s"(hv) : "s"(cfl), "s"(j * 64) : "memory");
        u32 bad = 0;
#pragma unroll
        for (int i = 0; i < 8; ++i) bad |= hv[i] ^ (ctag << 20);
        words = bad >> 20 ? ~0u : (hv[1] >> 9) & 0x3ff;
        hw = lane == 0 ? (hv[0] & 0xfffffu) | (hv[1] << 20) : lane == 1 ? (hv[2] & 0xfffffu) | (hv[3] << 20)
           : lane == 2 ? (hv[4] & 0xfffffu) | (hv[5] << 20) : (hv[6] & 0xfffffu) | (hv[7] << 20);
      };
#pragma unroll 1
      for (int j = j0; j < j1; ++j) {
        const u32 n = dr + j * NDC - JOFF, g = GH + (n >> 1), k = n & 1;
        announce(trel());
        bool dead = false, waited = false;
        const u64 tw = rt();
        for (u32 it = 1; words == ~0u; ++it) {
          headers_of(j);
          if (words != ~0u) { waited = it > 1; break; }
          announce(trel());
          if ((it & 3) == 0 && !allseen) poll();
          if ((it & 63) == 0 && (__hip_atomic_load(tmo, __ATOMIC_RELAXED, __HIP_MEMORY_SCOPE_AGENT) == call || rt() - tw > DEADLINE)) {
            if (lane == 0) __hip_atomic_store(tmo, call, __ATOMIC_RELAXED, __HIP_MEMORY_SCOPE_AGENT);
            dead = true;
            break;
          }
        }
        if (dead) { ctrl[59] = 1; break; }
        const u32 nraw = g + NT >= 256 ? n - 2 * (256 - NT - GH) : 0;
        const u32 off = PACE_T0 + (((n >> 1) * (8 * DT_PK) + k * (8 * DT_PK * 54 / 100) + dq * (4 * DT_PK / 7) + nraw * (4 * DT_PK * RAWX / 100)) >> 8);
        const u32 bytes = (k ? 4 * 1280 : 4 * 1536) + 4 * words;
        const u32 now = trel();
        u32 tk;
        while (true) {
          idle_until(base + off - OLEAD);
          u32 o;
          asm volatile("s_load_dword %0, %2, 0x0 glc\n\ts_load_dword %1, %3, 0x0 glc\n\ts_waitcnt lgkmcnt(0)"
                       : "=&s"(o), "=&s"(tk) : "s"(org), "s"(myws + W_TOK + di * 128) : "memory");
          const u32 nb = o - (u32)tstart <= trel() ? o - (u32)tstart : 0;
          const bool again = nb > base + 30 && nb + off > trel() + 2 * OLEAD;
          if (nb > base) base = nb;
          if (waited && now > base + off + GRACE) {
            base = trel() + OLEAD + 30 - off;
            move_origin(dq, tstart + base);
          } else if (!again) break;
          waited = false;
        }
        if (n >= WIN && (int)(tk - (call * 16 + j)) < 0) {
          u32 t2;
          do {
            asm volatile("s_load_dword %0, %1, 0x0 glc\n\ts_waitcnt lgkmcnt(0)" : "=s"(tk) : "s"(myws + W_TOK + di * 128) : "memory");
            t2 = trel();
          } while ((int)(tk - (call * 16 + j)) < 0 && t2 < base + off + 1000);
          if (t2 > base + off + TGRACE && n >= TSTART) {
            base = t2 - off;
            move_origin(dq, tstart + base);
          }
        }
        const u32 due = base + off;
        while (!__all((int)ctrl[57 + (lane & 1)] >= j - 1)) __builtin_amdgcn_s_sleep(1);
        if (lane < 4) ctrl[24 + (j & 1) * 4 + lane] = hw;
        const char* src = src0 + g * CSTRIDE + k * 4 * PIECE;
        const u32 l = __builtin_amdgcn_readfirstlane(L_STG + (j & 1) * 4 * PIECE);
        const u64 m5 = __ballot(5120 + lane * 16 < bytes), m6 = __ballot(6144 + lane * 16 < bytes), m7 = __ballot(7168 + lane * 16 < bytes);
        while (trel() < due) __builtin_amdgcn_s_sleep(1);
        ctrl[61] = j + 1;
#define G(o) "global_load_lds_dwordx4 %0, %1 offset:" #o " sc0 sc1\n\t"
#define G2(o) "global_load_lds_dwordx4 %0, %2 offset:" #o " sc0 sc1\n\t"
        asm volatile("s_mov_b32 m0, %3\n\ts_nop 3\n\t" G(0) G(1024) G(2048) G(3072)
                     "s_add_u32 m0, m0, 0x1000\n\ts_nop 0\n\t" G2(0)
                     "s_mov_b64 exec, %4\n\ts_cbranch_execz 1f\n\t" G2(1024)
                     "s_mov_b64 exec, %5\n\ts_cbranch_execz 1f\n\t" G2(2048)
                     "s_mov_b64 exec, %6\n\ts_cbranch_execz 1f\n\t" G2(3072)
                     "1:\n\ts_mov_b64 exec, -1"
                     :: "v"(lane * 16), "s"(src), "s"(src + 4096), "s"(l), "s"(m5), "s"(m6), "s"(m7) : "memory", "scc");
        const u32 tis = trel();
        words = ~0u;
        if (HPRE) headers_of(j + 1);
        if (FPOLL && !allseen && trel() < tis + FPOLL) poll();
        wait_vm<0>();
        asm volatile("s_waitcnt lgkmcnt(0)" ::: "memory");
        ctrl[56] = j + 1;
        if (lane == 0) {
          const u32 r2 = dr + WIN, wrap = r2 >= NDC;
          __hip_atomic_store((GAS u32*)(myws + W_TOK + ((r2 - wrap * NDC) * 7 + dq) * 128), call * 16 + j + wrap, __ATOMIC_RELAXED, __HIP_MEMORY_SCOPE_AGENT);
        }
      }
    }
    STAMP(9);
    PUT(0, tstart); PUT(1, h1); PUT(5, h5); PUT(6, h6); PUT(7, h7);
    while ((!allseen && ctrl[2] == 0) || tann != ~0u) idle_until(trel() + 200);
  } else {
    const int d = wv - 10;
    if (d == 0 && j1 && dr == 20) {
      GAS u32* hp = (GAS u32*)(p.peer[dq] + W_HELLO + (6 - dq) * 256);
      asm volatile("" : "+s"(hp));
      const u32 age = umin32((u32)rt() - (u32)tstart, 1023);
      if (lane == 0) __hip_atomic_store(hp, (call << 10) | age, __ATOMIC_RELAXED, __HIP_MEMORY_SCOPE_SYSTEM);
      const u32 ho = __builtin_amdgcn_readfirstlane(dq * 256);
      u32 hc, it = 0;
      u64 tn;
      do {
        asm volatile("s_load_dword %0, %1, %2 glc\n\ts_waitcnt lgkmcnt(0)" : "=&s"(hc) : "s"(myws + W_HELLO), "s"(ho) : "memory");
        tn = rt() - FLAT - (hc & 1023);
        ++it;
      } while ((int)(hc - (call << 10)) < 0 && __builtin_amdgcn_readfirstlane(ctrl[56] | ctrl[59]) == 0);
      move_origin(dq, (it > 1 && tn > tstart ? tn : tstart) + 100);
    }
    while ((u32)rt() - (u32)tstart < QUIET) __builtin_amdgcn_s_sleep(8);
    while (ctrl[51] == 0) __builtin_amdgcn_s_sleep(1);
    LAS char* scr = lds + L_DEC + d * DEC_BYTES;
#pragma unroll 1
    for (int j = j0; j < j1; ++j) {
      const u32 n = dr + j * NDC - JOFF, g = GH + (n >> 1), k = n & 1;
      u32 jb = p.order[g >> 2] >> (8 * (g & 3));
      asm volatile("" : "+s"(jb));
      const int jh = (jb >> 4) & 15, jm = jb & 15;
      char* out = myws + W_DATA + par + data_off(16 + 7 * jh + dq, 2 * jm + k) + 2 * d * PIECE;
      bool dead = false;
      while ((int)__builtin_amdgcn_readfirstlane(ctrl[61]) <= j && !(dead = __builtin_amdgcn_readfirstlane(ctrl[59]) != 0)) {
        if (DPOLL && d == 1 && j > j0 && !allseen) poll();
        __builtin_amdgcn_s_sleep(2);
      }
      if (dead) break;
      const u32x2 hh = *(volatile LAS u32x2*)(lds + L_CTRL + 96 + (j & 1) * 16 + d * 8);
      const u32 msg = L_STG + (j & 1) * 4 * PIECE, fix = k ? 1280 : 1536;
      u32x4 A[2], B[2], q0[2], q1[2];
      u32 h[2], C[2], aa[2], ab[2], a2[2], a3[2], a4[2], sh[2];
#pragma unroll
      for (int m = 0; m < 2; ++m) {
        h[m] = __builtin_amdgcn_readfirstlane(hh[m]);
        const bool raw = h[m] & ST_RAW;
        const u32 t2 = msg + 4 * fix + 4 * ((h[m] >> 13) & 0x3ff), t3 = t2 + (raw ? 256 : ((h[m] >> 5) & 0x7f) * 4), t4 = t3 + (raw ? 256 : ((h[m] >> 23) & 0x3f) * 4);
        aa[m] = msg + (2 * d + m) * fix + lane * 16;
        ab[m] = msg + (2 * d + m) * fix + 1024 + lane * (k ? 4 : 8);
        a2[m] = t2 + lane * 4; a3[m] = t3 + (raw ? lane : lane >> 1) * 4; a4[m] = t4 + lane * 4;
        sh[m] = raw ? 0 : (lane & 1) * 16;
        asm volatile("" : "+v"(aa[m]), "+v"(ab[m]), "+v"(a2[m]), "+v"(a3[m]), "+v"(a4[m]), "+v"(sh[m]));
      }
      while ((int)__builtin_amdgcn_readfirstlane(ctrl[56]) <= j) __builtin_amdgcn_s_sleep(DSLEEP);
      __builtin_amdgcn_fence(__ATOMIC_ACQUIRE, "workgroup");
#pragma unroll
      for (int m = 0; m < 2; ++m) {
        B[m].z = lds_ld(a2[m]);
        A[m] = *(const LAS u32x4*)(lds + aa[m]);
        B[m].w = lds_ld(a3[m]) >> sh[m];
      }
#pragma unroll
      for (int m = 0; m < 2; ++m) { C[m] = lds_ld(a4[m]); B[m].x = lds_ld(ab[m]); B[m].y = lds_ld(ab[m] + 4); }
      if (!(h[0] & h[1] & ST_RAW)) {
        if (k) decode_pair<true>(A, B, C, scr, lds + L_LUTD, h, lane, q0, q1);
        else decode_pair<false>(A, B, C, scr, lds + L_LUTD, h, lane, q0, q1);
      }
#pragma unroll
      for (int m = 0; m < 2; ++m) {
        if (h[m] & ST_RAW) { q0[m] = A[m]; q1[m] = B[m]; if (k) q1[m].y = C[m]; }
        st2_sys(out + m * PIECE, lane * 16, q0[m], q1[m]);
      }
      ctrl[57 + d] = j + 1;
      wait_vm<0>();
      if (lds_ticket(ctrl + 32 + j, lane) == 1) publish(16 + 7 * jh + dq, 2 * jm + k);
    }
    if (j1 && lds_ticket(ctrl + 60, lane) == 1) ctrl[5] = 0;
    STAMP(2);
  }
}

static void make_plan(Params& p) {
  unsigned char ord[256];
  int n = 0;
  for (int key = 0; n < 256; ++key)
    for (int m = 0; m < 16; ++m)
      for (int hc = 0; hc < 16; ++hc)
        if (SKA * hc + SKB * m + SKC * (m < SKCAP ? m : SKCAP) * (m < SKCAP ? m : SKCAP) / 16 == key) ord[n++] = (unsigned char)(hc << 4 | m);
  for (int i = 0; i < 64; ++i) p.order[i] = ord[4 * i] | ord[4 * i + 1] << 8 | ord[4 * i + 2] << 16 | (u32)ord[4 * i + 3] << 24;
  for (int b = 0; b < 256; ++b) {
    const int x = b & 7;
    auto row = [&](int l) { return 4 * (l & 3) + ((l & 1) ? 3 - (x >> 1) : (x >> 1)); };
    auto col = [&](int l) { return 2 * (l >> 2) + (((l >> 2) ^ x) & 1); };
    auto rnk = [&](int l) { const int i = row(l), j = col(l); return j < i ? i * i + j : j * j + j + i; };
    int lr = 0;
    for (int l = 0; l < 32; ++l) lr += rnk(l) < rnk(b >> 3);
    const int s = lr * 8 + x, g = (s + BROT) & 255;
    p.plan[b] = s | row(b >> 3) << 8 | col(b >> 3) << 12 | ord[g] << 16 | (u32)g << 24;
  }
}

extern "C" void kernel_launch(int rank, int nranks, ncclComm_t comm,
                              hipStream_t stream,
                              void* const* d_in, const int* in_sizes, int n_in,
                              void* d_out, int out_size,
                              const int* in_dtypes, int out_dtype,
                              void* const* d_out_all,
                              void* const* d_ws_all, size_t ws_bytes) {
  if (nranks != NG || (unsigned)rank >= (unsigned)NG || n_in < 4 || in_sizes[0] != MM * KL || in_sizes[1] != KL * NN ||
      out_size != MM * NN || ws_bytes < W_END) {
    fprintf(stderr, "kernel_launch: unexpected problem shape\n");
    return;
  }
  HIP_CHECK(hipFuncSetAttribute((const void*)fused_gemm_ag, hipFuncAttributeMaxDynamicSharedMemorySize, LDS_BYTES));
  Params p;
  p.own = (char*)d_ws_all[rank];
  for (int q = 0; q < 7; ++q) p.peer[q] = (char*)d_ws_all[(rank + 1 + q) % NG];
  make_plan(p);
  p.x = (const unsigned char*)d_in[0];
  p.w = (const unsigned char*)d_in[1];
  p.sx = (const float*)d_in[2];
  p.sw = (const float*)d_in[3];
  p.out = (float*)d_out;
  p.dbg = nullptr;
  hipLaunchKernelGGL(fused_gemm_ag, dim3(256), dim3(768), LDS_BYTES, stream, p);
  HIP_CHECK(hipGetLastError());
}
```
